```python
import numpy as np

kernel_name = "gpu_gemm_ar_m8192_k8192_n8192_f32_none"


ABSMAX_RTOL = 0.001

COMPUTE_DTYPE = "float32"
EPILOGUE = "none"
IN_QUANT = None
OUT_QUANT = None
IN_DTYPES = {"x": "float32", "w_mat": "float32"}
WORLD_SIZE = 8

def setup_inputs():

    x = np.random.randn(8192, 8192).astype(np.float32)
    w_mat = (np.random.randn(8192, 8192) / np.sqrt(8192)).astype(np.float32)
    return {"x": x, "w_mat": w_mat}

def _epilogue(y):
    return y

def reference(x, w_mat):
    y = (x @ w_mat).astype(np.float32)
    return _epilogue(y)
```

```cpp
#include <hip/hip_runtime.h>
#include <rccl/rccl.h>
#include <cstdio>
#include <cstdint>
#include <algorithm>
#include <vector>

#define HIP_CHECK(expr) do { hipError_t _e = (expr); if (_e != hipSuccess) { \
    fprintf(stderr, "HIP error %s at %s:%d\n", hipGetErrorString(_e), __FILE__, __LINE__); \
    return; } } while (0)

namespace {
#define LAS __attribute__((address_space(3)))
typedef _Float16 f16x8 __attribute__((ext_vector_type(8)));
typedef float f32x4 __attribute__((ext_vector_type(4)));
typedef unsigned u32x4 __attribute__((ext_vector_type(4)));

constexpr int NR = 8;
constexpr int MN = 8192;
constexpr int KSH = 1024;
constexpr int BM = 256, BK = 64, HALF = 128;
constexpr int HTB = HALF * BK * 2;
constexpr int NKT = NR * KSH / BK;
constexpr int KT_PER_RANK = KSH / BK;
constexpr int NPAN = MN / BM;
constexpr size_t MAT_BYTES = (size_t)MN * NR * KSH * 2;
constexpr size_t CTRL_BYTES = (size_t)1 << 20;
constexpr size_t WS_NEED = CTRL_BYTES + 4 * MAT_BYTES;
constexpr int STAGE_BYTES = 8 * HTB;
constexpr int LDS_BYTES = STAGE_BYTES + 64;
constexpr int GRID = 256;
constexpr int NPUSH = 8;
constexpr unsigned long long DEADLINE = 50000000ull;

constexpr size_t OFF_CALL = 0;
constexpr size_t OFF_TMO = 128;
constexpr size_t OFF_QUEUE = 1024;
constexpr size_t OFF_FLAG = 8192;
__host__ __device__ constexpr size_t flag_off(int src, int mat, int panel) { return OFF_FLAG + (size_t)((src * 2 + mat) * NPAN + panel) * 128; }

struct Params {
    const float* x; const float* w; float* out;
    char* ws[NR];
    int rank;
    unsigned short order[NR][128];
};

__device__ __forceinline__ int lds_byte(int r, int c) { const int st = (r >> 4) * 2 + (c >> 5), rr = r & 15, cc = c & 31, ob = rr * 64 + cc * 2; return st * 1024 + (ob ^ (((ob >> 9) & 1) << 5)); }
__device__ __forceinline__ void stage_rc(int b, int& R, int& C) { const int st = b / 1024, sb = b % 1024, swz = sb ^ (((sb >> 9) & 1) << 5); R = (st >> 1) * 16 + swz / 64; C = (st & 1) * 32 + (swz % 64) / 2; }

__device__ __forceinline__ unsigned pack2(float a, float b) {
    const _Float16 ha = (_Float16)a, hb = (_Float16)b;
    return (unsigned)__builtin_bit_cast(unsigned short, ha) | ((unsigned)__builtin_bit_cast(unsigned short, hb) << 16);
}

__device__ __forceinline__ void push_phase(const Params& p, int pw, unsigned seq) {
    const int tid = threadIdx.x;
    const size_t par = (size_t)(seq & 1) * 2 * MAT_BYTES;
    int R[2], C[2];
    stage_rc(tid * 16, R[0], C[0]); stage_rc(tid * 16 + 8192, R[1], C[1]);
    char* dst[NR];
#pragma unroll
    for (int q = 0; q < NR; ++q) dst[q] = p.ws[(p.rank + q) & (NR - 1)];
    for (int c = pw; c < 2 * NPAN; c += NPUSH) {
        const int mat = c & 1, panel = c >> 1;
        for (int hb2 = 0; hb2 < 2; ++hb2) {
            const int row0 = panel * BM + hb2 * HALF;
            for (int kt = 0; kt < KT_PER_RANK; kt += 2) {
                u32x4 v[4];
                if (mat == 0) {
                    f32x4 a[4][2];
#pragma unroll
                    for (int u = 0; u < 4; ++u) {
                        const float* s = p.x + (size_t)(row0 + R[u & 1]) * KSH + (kt + (u >> 1)) * BK + C[u & 1];
                        a[u][0] = *(const f32x4*)s; a[u][1] = *(const f32x4*)(s + 4);
                    }
#pragma unroll
                    for (int u = 0; u < 4; ++u) {
                        v[u].x = pack2(a[u][0][0], a[u][0][1]); v[u].y = pack2(a[u][0][2], a[u][0][3]);
                        v[u].z = pack2(a[u][1][0], a[u][1][1]); v[u].w = pack2(a[u][1][2], a[u][1][3]);
                    }
                } else {
                    float a[4][8];
#pragma unroll
                    for (int u = 0; u < 4; ++u) {
                        const float* s = p.w + (size_t)((kt + (u >> 1)) * BK + C[u & 1]) * MN + row0 + R[u & 1];
#pragma unroll
                        for (int e = 0; e < 8; ++e) a[u][e] = s[(size_t)e * MN];
                    }
#pragma unroll
                    for (int u = 0; u < 4; ++u) {
                        v[u].x = pack2(a[u][0], a[u][1]); v[u].y = pack2(a[u][2], a[u][3]);
                        v[u].z = pack2(a[u][4], a[u][5]); v[u].w = pack2(a[u][6], a[u][7]);
                    }
                }
#pragma unroll
                for (int u = 0; u < 4; ++u) {
                    const size_t off = CTRL_BYTES + par + (size_t)mat * MAT_BYTES +
                                       ((size_t)(panel * 2 + hb2) * NKT + p.rank * KT_PER_RANK + kt + (u >> 1)) * HTB + (u & 1) * 8192 + tid * 16;
#pragma unroll
                    for (int q = 0; q < NR; ++q) *(u32x4*)(dst[q] + off) = v[u];
                }
            }
        }
        asm volatile("s_waitcnt vmcnt(0)" ::: "memory");
        __syncthreads();
        if (tid < NR) {
            __builtin_amdgcn_fence(__ATOMIC_RELEASE, "");
            asm volatile("s_waitcnt vmcnt(0)" ::: "memory");
            unsigned* f = (unsigned*)(p.ws[(p.rank + tid) & (NR - 1)] + flag_off(p.rank, mat, panel));
            __hip_atomic_store(f, seq, __ATOMIC_RELAXED, __HIP_MEMORY_SCOPE_SYSTEM);
        }
    }
}

__device__ __forceinline__ int next_tile(const Params& p, unsigned seq, unsigned xcc, int lane, bool& dead) {
    char* myws = p.ws[p.rank];
    int code = -1;
    if (lane == 0) {
        for (int s = 0; s < NR && code < 0; ++s) {
            const unsigned q = (xcc + s) & (NR - 1);
            unsigned long long* qp = (unsigned long long*)(myws + OFF_QUEUE + q * 128);
            unsigned long long old = __hip_atomic_load(qp, __ATOMIC_RELAXED, __HIP_MEMORY_SCOPE_AGENT);
            for (;;) {
                const unsigned cnt = ((unsigned)(old >> 32) == seq) ? (unsigned)old : 0u;
                if (cnt >= 128u) break;
                const unsigned long long nw = ((unsigned long long)seq << 32) | (cnt + 1u);
                if (__hip_atomic_compare_exchange_strong(qp, &old, nw, __ATOMIC_RELAXED, __ATOMIC_RELAXED, __HIP_MEMORY_SCOPE_AGENT)) { code = p.order[q][cnt]; break; }
            }
        }
    }
    code = __builtin_amdgcn_readfirstlane(code);
    if (code >= 0) {
        const int pm = code >> 8, pn = code & 255;
        const unsigned* f = (const unsigned*)(myws + flag_off((lane >> 1) & 7, lane & 1, (lane & 1) ? pn : pm));
        unsigned* tmo = (unsigned*)(myws + OFF_TMO);
        const unsigned long long t0 = __builtin_amdgcn_s_memrealtime();
        for (;;) {
            const unsigned v = __hip_atomic_load(f, __ATOMIC_RELAXED, __HIP_MEMORY_SCOPE_SYSTEM);
            if (__all((int)(v - seq) >= 0)) break;
            const unsigned tv = __hip_atomic_load(tmo, __ATOMIC_RELAXED, __HIP_MEMORY_SCOPE_AGENT);
            if (__builtin_amdgcn_readfirstlane(tv) == seq || __builtin_amdgcn_s_memrealtime() - t0 > DEADLINE) {
                if (lane == 0) __hip_atomic_store(tmo, seq, __ATOMIC_RELAXED, __HIP_MEMORY_SCOPE_AGENT);
                dead = true; break;
            }
            __builtin_amdgcn_s_sleep(2);
        }
        __builtin_amdgcn_fence(__ATOMIC_ACQUIRE, "");
        asm volatile("s_waitcnt vmcnt(0)" ::: "memory");
    }
    return code;
}

__device__ __forceinline__ void gemm_phase(const Params& p, LAS unsigned char* lds, unsigned seq) {
    const int tid = threadIdx.x, wid = __builtin_amdgcn_readfirstlane(tid >> 6), lane = tid & 63, wr = wid >> 2, wc = wid & 3, fr = lane & 15, fq = lane >> 4;
    const unsigned xcc = (unsigned)__builtin_amdgcn_s_getreg((3 << 11) | 20) & 7u;
    volatile LAS int* bc = (volatile LAS int*)(lds + STAGE_BYTES + 16);
    const char* Abase = p.ws[p.rank] + CTRL_BYTES + (size_t)(seq & 1) * 2 * MAT_BYTES;
    const char* Bbase = Abase + MAT_BYTES;
    constexpr size_t kstep = HTB;
    constexpr size_t hstep = (size_t)NKT * HTB;
    constexpr size_t tstep = 2 * hstep;
    const unsigned voff = (unsigned)tid * 16u;
    const unsigned ldsw = (unsigned)wid * 1024u;
    const int aoff = lds_byte(wr * 64 + fr, fq * 8), boff = lds_byte(wc * 32 + fr, fq * 8);
    bool dead = false;
#define G_SA(b, h) (((b) * 2 + (h)) * HTB)
#define G_SB(b, h) ((4 + (b) * 2 + (h)) * HTB)
#define G_STAGE(bufoff, gbase) do { _Pragma("unroll") for (int _i = 0; _i < 2; ++_i) \
        __builtin_amdgcn_global_load_lds((const unsigned*)((const char*)(gbase) + voff + _i * 8192), (LAS unsigned*)(lds + (bufoff) + ldsw + _i * 8192), 16, 0, 0); } while (0)
#define G_LDA(dst, b, h) do { _Pragma("unroll") for (int m = 0; m < 4; ++m) _Pragma("unroll") for (int k = 0; k < 2; ++k) dst[m][k] = *(const LAS f16x8*)(lds + G_SA(b, h) + aoff + m * 2048 + k * 1024); } while (0)
#define G_LDB(dst, b, h) do { _Pragma("unroll") for (int n = 0; n < 2; ++n) _Pragma("unroll") for (int k = 0; k < 2; ++k) dst[n][k] = *(const LAS f16x8*)(lds + G_SB(b, h) + boff + n * 2048 + k * 1024); } while (0)
#define G_MMA(ai, bj, At, Bt) do { __builtin_amdgcn_s_setprio(1); _Pragma("unroll") for (int m = 0; m < 4; ++m) _Pragma("unroll") for (int n = 0; n < 2; ++n) _Pragma("unroll") for (int k = 0; k < 2; ++k) \
        acc[ai][bj][m][n] = __builtin_amdgcn_mfma_f32_16x16x32_f16(Bt[n][k], At[m][k], acc[ai][bj][m][n], 0, 0, 0); __builtin_amdgcn_s_setprio(0); } while (0)
#define G_WAIT_V(n) asm volatile("s_waitcnt vmcnt(" #n ")" ::: "memory")
#define G_WAIT_L(n) asm volatile("s_waitcnt lgkmcnt(" #n ")" ::: "memory")
#define G_BAR __builtin_amdgcn_s_barrier()
#define G_SCHED __builtin_amdgcn_sched_barrier(0)
#define G_NEXT(codevar) do { if (wid == 0) { bool _d = false; const int _c = next_tile(p, seq, xcc, lane, _d); if (lane == 0) { bc[0] = _c; if (_d) bc[1] = 1; } asm volatile("s_waitcnt lgkmcnt(0)" ::: "memory"); } \
        asm volatile("" ::: "memory"); G_BAR; asm volatile("" ::: "memory"); codevar = __builtin_amdgcn_readfirstlane(bc[0]); } while (0)

    if (tid == 0) { bc[0] = -1; bc[1] = 0; }
    __syncthreads();
    int cur;
    G_NEXT(cur);
    if (cur < 0) return;
    f32x4 acc[2][2][4][2];
#pragma unroll
    for (int a = 0; a < 2; ++a)
#pragma unroll
        for (int b = 0; b < 2; ++b)
#pragma unroll
            for (int m = 0; m < 4; ++m)
#pragma unroll
                for (int n = 0; n < 2; ++n) acc[a][b][m][n] = (f32x4){0.f, 0.f, 0.f, 0.f};
    f16x8 At[4][2], B0[2][2], B1[2][2];
    const char* cA = Abase + (size_t)(cur >> 8) * tstep; const char* cB = Bbase + (size_t)(cur & 255) * tstep;
    G_STAGE(G_SB(0, 0), cB); G_STAGE(G_SA(0, 0), cA); G_STAGE(G_SB(0, 1), cB + hstep); G_STAGE(G_SA(0, 1), cA + hstep);
    if (wr == 1) G_BAR;
    G_WAIT_V(4); G_BAR;
    G_STAGE(G_SB(1, 0), cB + kstep); G_STAGE(G_SA(1, 0), cA + kstep); G_STAGE(G_SB(1, 1), cB + hstep + kstep);
    G_WAIT_V(6); G_BAR;
    for (;;) {
        int nxt = -1; const char* nA = cA; const char* nB = cB;
        for (int t = 0; t < NKT; t += 2) {
            const bool last = (t == NKT - 2);
            if (last) {
                G_NEXT(nxt);
                if (nxt >= 0) { nA = Abase + (size_t)(nxt >> 8) * tstep; nB = Bbase + (size_t)(nxt & 255) * tstep; }
            }
            const char* a1 = cA + (size_t)(t + 1) * kstep;
            const char* a2 = last ? nA : cA + (size_t)(t + 2) * kstep; const char* b2 = last ? nB : cB + (size_t)(t + 2) * kstep;
            const char* a3 = a2 + kstep; const char* b3 = b2 + kstep;
            G_LDB(B0, 0, 0); G_SCHED; G_LDA(At, 0, 0); G_STAGE(G_SA(1, 1), a1 + hstep);
            G_WAIT_L(8); G_BAR; G_WAIT_L(0); G_MMA(0, 0, At, B0); G_BAR; G_SCHED;
            G_LDB(B1, 0, 1); G_STAGE(G_SB(0, 0), b2);
            G_BAR; G_WAIT_L(0); G_MMA(0, 1, At, B1); G_BAR;
            G_LDA(At, 0, 1); G_STAGE(G_SA(0, 0), a2);
            G_BAR; G_WAIT_L(0); G_MMA(1, 0, At, B0); G_BAR; G_SCHED;
            G_STAGE(G_SB(0, 1), b2 + hstep);
            G_WAIT_V(6); G_BAR; G_MMA(1, 1, At, B1); G_BAR;
            G_LDB(B0, 1, 0); G_SCHED; G_LDA(At, 1, 0); G_STAGE(G_SA(0, 1), a2 + hstep);
            G_WAIT_L(8); G_BAR; G_WAIT_L(0); G_MMA(0, 0, At, B0); G_BAR; G_SCHED;
            G_LDB(B1, 1, 1); G_STAGE(G_SB(1, 0), b3);
            G_BAR; G_WAIT_L(0); G_MMA(0, 1, At, B1); G_BAR;
            G_LDA(At, 1, 1); G_STAGE(G_SA(1, 0), a3);
            G_BAR; G_WAIT_L(0); G_MMA(1, 0, At, B0); G_BAR; G_SCHED;
            G_STAGE(G_SB(1, 1), b3 + hstep);
            G_WAIT_V(6); G_BAR; G_MMA(1, 1, At, B1); G_BAR;
        }
        {
            dead = (bc[1] != 0);
            const float qn = __builtin_nanf("");
            const int row0 = (cur >> 8) * BM + wr * 64 + fr, col0 = (cur & 255) * BM + wc * 32 + 4 * fq;
#pragma unroll
            for (int ai = 0; ai < 2; ++ai)
#pragma unroll
                for (int m = 0; m < 4; ++m) {
                    float* rowp = p.out + (size_t)(row0 + ai * HALF + m * 16) * MN + col0;
#pragma unroll
                    for (int bj = 0; bj < 2; ++bj)
#pragma unroll
                        for (int n = 0; n < 2; ++n) *(f32x4*)(rowp + bj * HALF + n * 16) = dead ? (f32x4){qn, qn, qn, qn} : acc[ai][bj][m][n];
                }
        }
        if (nxt < 0) break;
#pragma unroll
        for (int a = 0; a < 2; ++a)
#pragma unroll
            for (int b = 0; b < 2; ++b)
#pragma unroll
                for (int m = 0; m < 4; ++m)
#pragma unroll
                    for (int n = 0; n < 2; ++n) acc[a][b][m][n] = (f32x4){0.f, 0.f, 0.f, 0.f};
        cur = nxt; cA = nA; cB = nB;
    }
    G_WAIT_V(0);
    if (wr == 0) G_BAR;
    G_BAR;
}

__global__ __launch_bounds__(512, 2) void fused_kernel(Params p) {
    extern __shared__ __attribute__((aligned(16))) unsigned char shm[];
    LAS unsigned char* lds = (LAS unsigned char*)shm;
    volatile LAS unsigned* sq = (volatile LAS unsigned*)(lds + STAGE_BYTES);
    if (threadIdx.x == 0) {
        const unsigned t = __hip_atomic_fetch_add((unsigned*)(p.ws[p.rank] + OFF_CALL), 1u, __ATOMIC_RELAXED, __HIP_MEMORY_SCOPE_AGENT);
        sq[0] = t / GRID + 1u;
    }
    __syncthreads();
    const unsigned seq = __builtin_amdgcn_readfirstlane(sq[0]);
    if ((int)blockIdx.x < NPUSH) push_phase(p, (int)blockIdx.x, seq);
    gemm_phase(p, lds, seq);
}
}

extern "C" void kernel_launch(int rank, int nranks, ncclComm_t comm,
                              hipStream_t stream,
                              void* const* d_in, const int* in_sizes, int n_in,
                              void* d_out, int out_size,
                              const int* in_dtypes, int out_dtype,
                              void* const* d_out_all,
                              void* const* d_ws_all, size_t ws_bytes) {
    if (nranks != NR || ws_bytes < WS_NEED || n_in < 2) { fprintf(stderr, "unsupported configuration\n"); return; }
    static bool init = false;
    static Params P;
    if (!init) {
        HIP_CHECK(hipFuncSetAttribute((const void*)fused_kernel, hipFuncAttributeMaxDynamicSharedMemorySize, LDS_BYTES));
        for (int x = 0; x < NR; ++x) {
            std::vector<std::pair<int, int>> v;
            for (int i = 0; i < 16; ++i) for (int j = 0; j < 8; ++j) {
                const int pm = 2 * i + (x & 1), pn = 4 * j + (x >> 1);
                const int key = std::max(2 * pm, 2 * pn + 1);
                v.push_back({key * 4096 + (pm + pn) * 64 + pm, pm << 8 | pn});
            }
            std::sort(v.begin(), v.end());
            for (int k = 0; k < 128; ++k) P.order[x][k] = (unsigned short)v[k].second;
        }
        init = true;
    }
    P.x = (const float*)d_in[0]; P.w = (const float*)d_in[1]; P.out = (float*)d_out; P.rank = rank;
    for (int r = 0; r < NR; ++r) P.ws[r] = (char*)d_ws_all[r];
    hipLaunchKernelGGL(fused_kernel, dim3(GRID), dim3(512), LDS_BYTES, stream, P);
    HIP_CHECK(hipGetLastError());
}
```

```cpp
#include <hip/hip_runtime.h>
#include <rccl/rccl.h>
#include <cstdio>
#include <cstdint>
#include <algorithm>
#include <vector>

#define HIP_CHECK(expr) do { hipError_t _e = (expr); if (_e != hipSuccess) { \
    fprintf(stderr, "HIP error %s at %s:%d\n", hipGetErrorString(_e), __FILE__, __LINE__); \
    return; } } while (0)

namespace {
#define LAS __attribute__((address_space(3)))
typedef _Float16 f16x8 __attribute__((ext_vector_type(8)));
typedef float f32x4 __attribute__((ext_vector_type(4)));
typedef unsigned u32x4 __attribute__((ext_vector_type(4)));

constexpr int NR = 8;
constexpr int MN = 8192;
constexpr int KSH = 1024;
constexpr int BM = 256, BK = 64, HALF = 128;
constexpr int HTB = HALF * BK * 2;
constexpr int NKT = NR * KSH / BK;
constexpr int KT_PER_RANK = KSH / BK;
constexpr int NPAN = MN / BM;
constexpr int NG = 4;
constexpr size_t MAT_BYTES = (size_t)MN * NR * KSH * 2;
constexpr size_t CTRL_BYTES = (size_t)1 << 20;
#define NSPLIT 12
#define NSH 12
static_assert(NSH % 4 == 0 && NSH <= 12, "NSH");
constexpr int SHBASE = NPAN - NSH;
constexpr int NOWN = NSH * NPAN / NR;
__host__ __device__ constexpr int sh_owner(int pm, int pn) { return ((pm - SHBASE) * 2 + (pn >> 4)) & (NR - 1); }
constexpr int QTILES = (SHBASE * NPAN + NOWN) / NR;
constexpr int NWHOLE = QTILES - NSPLIT;
constexpr int QLEN = NWHOLE + NSPLIT * NG;
constexpr size_t TILE_BYTES = (size_t)BM * BM * 4;
constexpr size_t OFF_PART = CTRL_BYTES + 4 * MAT_BYTES;
constexpr size_t WS_NEED = OFF_PART + (size_t)NR * NSPLIT * NG * TILE_BYTES;
constexpr int STAGE_BYTES = 8 * HTB;
constexpr int LDS_BYTES = STAGE_BYTES + 64;
constexpr int GRID = 256;
#define NPUSH 32
constexpr unsigned long long DEADLINE = 50000000ull;

constexpr size_t OFF_CALL = 0;
constexpr size_t OFF_TMO = 128;
constexpr size_t OFF_QUEUE = 1024;
constexpr size_t OFF_CENSUS = 4096;
constexpr size_t OFF_FLAG = 8192;
constexpr size_t OFF_GRP = 64 * 1024;
constexpr size_t OFF_CNT = 256 * 1024;
constexpr size_t OFF_DONE = 384 * 1024;
constexpr size_t OFF_OUTF = 448 * 1024;
__host__ __device__ constexpr size_t flag_off(int src, int reg, int g) { return OFF_FLAG + (size_t)((src * 3 + reg) * NG + g) * 128; }
constexpr unsigned FLAG_PER_CALL = NPUSH;

#define STAMP(tag) do { } while (0)
#define REL_SYS() __builtin_amdgcn_fence(__ATOMIC_RELEASE, "")
#define STMODE 1
#define ST_REMOTE(ptr, val) asm volatile("global_store_dwordx4 %0, %1, off sc0 sc1" :: "v"(ptr), "v"(val) : "memory")
struct Params {
    const float* x; const float* w; float* out;
    char* ws[NR];
    float* outs[NR];
    int rank;
    unsigned short order[NR][128];
    unsigned short own[64];
};

__device__ __forceinline__ int lds_byte(int r, int c) { const int st = (r >> 4) * 2 + (c >> 5), rr = r & 15, cc = c & 31, ob = rr * 64 + cc * 2; return st * 1024 + (ob ^ (((ob >> 9) & 1) << 5)); }
__device__ __forceinline__ void stage_rc(int b, int& R, int& C) { const int st = b / 1024, sb = b % 1024, swz = sb ^ (((sb >> 9) & 1) << 5); R = (st >> 1) * 16 + swz / 64; C = (st & 1) * 32 + (swz % 64) / 2; }

__device__ __forceinline__ unsigned pack2(float a, float b) {
    const _Float16 ha = (_Float16)a, hb = (_Float16)b;
    return (unsigned)__builtin_bit_cast(unsigned short, ha) | ((unsigned)__builtin_bit_cast(unsigned short, hb) << 16);
}

__device__ __forceinline__ unsigned seq_inc(unsigned long long* q, unsigned seq, unsigned add) {
    unsigned long long old = __hip_atomic_load(q, __ATOMIC_RELAXED, __HIP_MEMORY_SCOPE_AGENT);
    for (;;) {
        const unsigned cnt = ((unsigned)(old >> 32) == seq) ? (unsigned)old : 0u;
        if (__hip_atomic_compare_exchange_strong(q, &old, ((unsigned long long)seq << 32) | (cnt + add), __ATOMIC_RELAXED, __ATOMIC_RELAXED, __HIP_MEMORY_SCOPE_AGENT)) return cnt;
    }
}
__device__ __forceinline__ unsigned seq_get(unsigned long long* q, unsigned seq) {
    const unsigned long long v = __hip_atomic_load(q, __ATOMIC_RELAXED, __HIP_MEMORY_SCOPE_AGENT);
    return ((unsigned)(v >> 32) == seq) ? (unsigned)v : 0u;
}

__device__ __forceinline__ void push_phase(const Params& p, int pw, unsigned seq, int& nst) {
    const int tid = threadIdx.x;
    const size_t par = (size_t)(seq & 1) * 2 * MAT_BYTES;
    int R[2], C[2];
    stage_rc(tid * 16, R[0], C[0]); stage_rc(tid * 16 + 8192, R[1], C[1]);
    const int whb = tid >> 8, wl = tid & 255;
    const int wkg = wl & 7, wng = wl >> 3;
    char* dst[NR];
#pragma unroll
    for (int q = 0; q < NR; ++q) dst[q] = p.ws[(p.rank + q) & (NR - 1)];
    char* myws = p.ws[p.rank];
    const unsigned xcc = (unsigned)__builtin_amdgcn_s_getreg((3 << 11) | 20) & 7u;
    unsigned ngrp = 0;
    if (tid == 0) {
        seq_inc((unsigned long long*)(myws + OFF_CENSUS + xcc * 64), seq, 1u);
        seq_inc((unsigned long long*)(myws + OFF_CENSUS + 512), seq, 1u);
        const unsigned long long t0 = __builtin_amdgcn_s_memrealtime();
        while (seq_get((unsigned long long*)(myws + OFF_CENSUS + 512), seq) < (unsigned)NPUSH) {
            if (__builtin_amdgcn_s_memrealtime() - t0 > DEADLINE) { __hip_atomic_store((unsigned*)(myws + OFF_TMO), seq, __ATOMIC_RELAXED, __HIP_MEMORY_SCOPE_AGENT); break; }
            __builtin_amdgcn_s_sleep(1);
        }
        ngrp = seq_get((unsigned long long*)(myws + OFF_CENSUS + xcc * 64), seq);
    }
    for (int step = 0; step < 3 * NG; ++step) {
        const int reg = step >> 2, g = step & 3;
        const int nitem = (reg == 0 ? 2 * 16 + NSH : reg == 1 ? 16 : SHBASE - 16) * 4;
        for (int it = pw; it < nitem; it += NPUSH) {
            const int pi = it >> 2, kt = g * 4 + (it & 3);
            int mat, j; unsigned mask = 0xffu;
            if (reg == 0) { if (pi < 16) { mat = 0; j = pi; } else if (pi < 32) { mat = 1; j = pi - 16; }
                            else { mat = 0; j = SHBASE + pi - 32; mask = (1u << sh_owner(j, 0)) | (1u << sh_owner(j, 16)); } }
            else if (reg == 1) { mat = 1; j = 16 + pi; }
            else { mat = 0; j = 16 + pi; }
            const size_t koff = CTRL_BYTES + par + (size_t)(p.rank * KT_PER_RANK + kt) * HTB;
            u32x4 o[4]; size_t off[4];
            if (mat == 0) {
                f32x4 a[4][2];
#pragma unroll
                for (int v = 0; v < 4; ++v) {
                    const float* sp = p.x + (size_t)(j * BM + (v >> 1) * HALF + R[v & 1]) * KSH + kt * BK + C[v & 1];
                    a[v][0] = *(const f32x4*)sp; a[v][1] = *(const f32x4*)(sp + 4);
                }
#pragma unroll
                for (int v = 0; v < 4; ++v) {
                    o[v].x = pack2(a[v][0][0], a[v][0][1]); o[v].y = pack2(a[v][0][2], a[v][0][3]);
                    o[v].z = pack2(a[v][1][0], a[v][1][1]); o[v].w = pack2(a[v][1][2], a[v][1][3]);
                    off[v] = koff + (size_t)(j * 2 + (v >> 1)) * NKT * HTB + (v & 1) * 8192 + tid * 16;
                }
            } else {
                f32x4 bq[8];
                const float* sp = p.w + (size_t)(kt * BK + wkg * 8) * MN + j * BM + whb * HALF + wng * 4;
#pragma unroll
                for (int e = 0; e < 8; ++e) bq[e] = *(const f32x4*)(sp + (size_t)e * MN);
#pragma unroll
                for (int e = 0; e < 4; ++e) {
                    o[e].x = pack2(bq[0][e], bq[1][e]); o[e].y = pack2(bq[2][e], bq[3][e]);
                    o[e].z = pack2(bq[4][e], bq[5][e]); o[e].w = pack2(bq[6][e], bq[7][e]);
                    off[e] = koff + MAT_BYTES + (size_t)(j * 2 + whb) * NKT * HTB + lds_byte(wng * 4 + e, wkg * 8);
                }
            }
#pragma unroll
            for (int q = 0; q < NR; ++q) {
                if (!((mask >> ((p.rank + q) & (NR - 1))) & 1u)) continue;
#pragma unroll
                for (int v = 0; v < 4; ++v) ST_REMOTE(dst[q] + off[v], o[v]);
            }
        }
        asm volatile("s_waitcnt vmcnt(0)" ::: "memory");
        __builtin_amdgcn_s_barrier();
        if (tid == 0) {
            const unsigned old = seq_inc((unsigned long long*)(myws + OFF_GRP + (size_t)(xcc * 16 + step) * 64), seq, 1u);
            if (old + 1u == ngrp) {
                REL_SYS();
                asm volatile("s_waitcnt vmcnt(0)" ::: "memory");
#pragma unroll
                for (int q = 0; q < NR; ++q)
                    __hip_atomic_fetch_add((unsigned*)(dst[q] + flag_off(p.rank, reg, g)), ngrp, __ATOMIC_RELAXED, __HIP_MEMORY_SCOPE_SYSTEM);
            }
        }
        STAMP(0x1000000 + step);
    }
}

__device__ __forceinline__ void forward_phase(const Params& p, LAS unsigned char* lds, int pw, unsigned seq, int& nst) {
    const int tid = threadIdx.x;
    char* myws = p.ws[p.rank];
    unsigned* tmo = (unsigned*)(myws + OFF_TMO);
    for (int i = pw; i < NOWN; i += NPUSH) {
        if (tid == 0) {
            const unsigned* d = (const unsigned*)(myws + OFF_DONE + (size_t)i * 64);
            const unsigned long long t0 = __builtin_amdgcn_s_memrealtime();
            while ((int)(__hip_atomic_load(d, __ATOMIC_RELAXED, __HIP_MEMORY_SCOPE_AGENT) - seq * 8u) < 0) {
                if (__hip_atomic_load(tmo, __ATOMIC_RELAXED, __HIP_MEMORY_SCOPE_AGENT) == seq || __builtin_amdgcn_s_memrealtime() - t0 > DEADLINE) {
                    __hip_atomic_store(tmo, seq, __ATOMIC_RELAXED, __HIP_MEMORY_SCOPE_AGENT); break; }
                __builtin_amdgcn_s_sleep(16);
            }
            __builtin_amdgcn_fence(__ATOMIC_ACQUIRE, "agent");
            asm volatile("s_waitcnt vmcnt(0)" ::: "memory");
        }
        __syncthreads();
        const int t = p.own[i];
        const size_t base = ((size_t)(t & 31) * BM) * MN + (size_t)((t >> 5) & 31) * BM;
        for (int r0 = 0; r0 < BM; r0 += 32) {
            f32x4 v[4]; size_t off[4];
#pragma unroll
            for (int u = 0; u < 4; ++u) { off[u] = base + (size_t)(r0 + u * 8 + (tid >> 6)) * MN + (tid & 63) * 4; v[u] = *(const f32x4*)(p.out + off[u]); }
#pragma unroll
            for (int q = 1; q < NR; ++q) {
                float* o = p.outs[(p.rank + q) & (NR - 1)];
#pragma unroll
                for (int u = 0; u < 4; ++u) ST_REMOTE(o + off[u], v[u]);
            }
        }
        asm volatile("s_waitcnt vmcnt(0)" ::: "memory");
        __syncthreads();
        if (tid == 0) {
            REL_SYS();
            asm volatile("s_waitcnt vmcnt(0)" ::: "memory");
#pragma unroll
            for (int q = 1; q < NR; ++q)
                __hip_atomic_fetch_add((unsigned*)(p.ws[(p.rank + q) & (NR - 1)] + OFF_OUTF + (size_t)p.rank * 128), 1u, __ATOMIC_RELAXED, __HIP_MEMORY_SCOPE_SYSTEM);
        }
        STAMP(0x4000000 + i);
    }
    if (pw == 0 && tid < 64) {
        const int src = tid & 7;
        const unsigned* f = (const unsigned*)(myws + OFF_OUTF + (size_t)src * 128);
        const unsigned want = seq * (unsigned)NOWN;
        const unsigned long long t0 = __builtin_amdgcn_s_memrealtime();
        bool dead = false;
        for (;;) {
            const unsigned v = __hip_atomic_load(f, __ATOMIC_RELAXED, __HIP_MEMORY_SCOPE_SYSTEM);
            if (__all(src == p.rank || (int)(v - want) >= 0)) break;
            const unsigned tv = __hip_atomic_load(tmo, __ATOMIC_RELAXED, __HIP_MEMORY_SCOPE_AGENT);
            if (__builtin_amdgcn_readfirstlane(tv) == seq || __builtin_amdgcn_s_memrealtime() - t0 > DEADLINE) { dead = true; break; }
            __builtin_amdgcn_s_sleep(16);
        }
        __builtin_amdgcn_fence(__ATOMIC_ACQUIRE, "");
        if (dead && tid == 0) { __hip_atomic_store(tmo, seq, __ATOMIC_RELAXED, __HIP_MEMORY_SCOPE_AGENT); p.out[0] = __builtin_nanf(""); }
        STAMP(0x5000000);
    }
}

__device__ __forceinline__ void wait_group(const Params& p, unsigned seq, int code, int g, int lane, bool& dead) {
    char* myws = p.ws[p.rank];
    const int pm = code & 255, pn = (code >> 8) & 255;
    const int reg = (lane & 1) ? (pn < NPAN / 2 ? 0 : 1) : ((pm < NPAN / 2 || pm >= SHBASE) ? 0 : 2);
    const unsigned* f = (const unsigned*)(myws + flag_off((lane >> 1) & 7, reg, g));
    const unsigned want = seq * FLAG_PER_CALL;
    unsigned* tmo = (unsigned*)(myws + OFF_TMO);
    const unsigned long long t0 = __builtin_amdgcn_s_memrealtime();
    for (;;) {
        const unsigned v = __hip_atomic_load(f, __ATOMIC_RELAXED, __HIP_MEMORY_SCOPE_SYSTEM);
        if (__all((int)(v - want) >= 0)) break;
        const unsigned tv = __hip_atomic_load(tmo, __ATOMIC_RELAXED, __HIP_MEMORY_SCOPE_AGENT);
        if (__builtin_amdgcn_readfirstlane(tv) == seq || __builtin_amdgcn_s_memrealtime() - t0 > DEADLINE) {
            if (lane == 0) __hip_atomic_store(tmo, seq, __ATOMIC_RELAXED, __HIP_MEMORY_SCOPE_AGENT);
            dead = true; break;
        }
        __builtin_amdgcn_s_sleep(16);
    }
    __builtin_amdgcn_fence(__ATOMIC_ACQUIRE, "wavefront");
    asm volatile("s_waitcnt vmcnt(0)" ::: "memory");
}

__device__ __forceinline__ int draw_unit(const Params& p, unsigned seq, unsigned xcc, int lane) {
    char* myws = p.ws[p.rank];
    int code = -1;
    if (lane == 0) {
        for (int s = 0; s < NR && code < 0; ++s) {
            const unsigned q = (xcc + s) & (NR - 1);
            unsigned long long* qp = (unsigned long long*)(myws + OFF_QUEUE + q * 128);
            unsigned long long old = __hip_atomic_load(qp, __ATOMIC_RELAXED, __HIP_MEMORY_SCOPE_AGENT);
            for (;;) {
                const unsigned cnt = ((unsigned)(old >> 32) == seq) ? (unsigned)old : 0u;
                if (cnt >= (unsigned)QLEN) break;
                const unsigned long long nw = ((unsigned long long)seq << 32) | (cnt + 1u);
                if (__hip_atomic_compare_exchange_strong(qp, &old, nw, __ATOMIC_RELAXED, __ATOMIC_RELAXED, __HIP_MEMORY_SCOPE_AGENT)) {
                    if (cnt < (unsigned)NWHOLE) { const int t = p.order[q][cnt]; code = (t & 31) | (((t >> 5) & 31) << 8); if (t >> 10) code |= (1 << 19) | (((t >> 10) - 1) << 20); }
                    else { const int j = (int)cnt - NWHOLE, ti = j % NSPLIT, kq = j / NSPLIT; const int t = p.order[q][NWHOLE + ti];
                           code = (t & 31) | (((t >> 5) & 31) << 8) | (kq << 16) | (1 << 18) | (((int)q * NSPLIT + ti) << 20); }
                    break;
                }
            }
        }
    }
    return __builtin_amdgcn_readfirstlane(code);
}

__device__ __forceinline__ void gemm_phase(const Params& p, LAS unsigned char* lds, unsigned seq, int& nst) {
    const int tid = threadIdx.x, wid = __builtin_amdgcn_readfirstlane(tid >> 6), lane = tid & 63, wr = wid >> 2, wc = wid & 3, fr = lane & 15, fq = lane >> 4;
    const unsigned xcc = (unsigned)__builtin_amdgcn_s_getreg((3 << 11) | 20) & 7u;
    volatile LAS int* bc = (volatile LAS int*)(lds + STAGE_BYTES + 16);
    const char* Abase = p.ws[p.rank] + CTRL_BYTES + (size_t)(seq & 1) * 2 * MAT_BYTES;
    const char* Bbase = Abase + MAT_BYTES;
    constexpr size_t hstep = (size_t)NKT * HTB;
    constexpr size_t tstep = 2 * hstep;
    const unsigned voff = (unsigned)tid * 16u;
    const unsigned ldsw = (unsigned)wid * 1024u;
    const int aoff = lds_byte(wr * 64 + fr, fq * 8), boff = lds_byte(wc * 32 + fr, fq * 8);
    const int rank = p.rank;
    bool dead = false;
#define KOFF(T) ((size_t)((((rank + (((T) >> 2) & 7)) & 7) << 4) | (((T) >> 5) << 2) | ((T) & 3)) * HTB)
#define U_T0(c) ((((c) >> 18) & 1) * (((c) >> 16) & 3) * 32)
#define U_NT(c) ((((c) >> 18) & 1) ? 32 : NKT)
#define G_SA(b, h) (((b) * 2 + (h)) * HTB)
#define G_SB(b, h) ((4 + (b) * 2 + (h)) * HTB)
#define G_STAGE(bufoff, gbase) do { _Pragma("unroll") for (int _i = 0; _i < 2; ++_i) \
        __builtin_amdgcn_global_load_lds((const unsigned*)((const char*)(gbase) + voff + _i * 8192), (LAS unsigned*)(lds + (bufoff) + ldsw + _i * 8192), 16, 0, 0); } while (0)
#define G_LDA(dst, b, h) do { _Pragma("unroll") for (int m = 0; m < 4; ++m) _Pragma("unroll") for (int k = 0; k < 2; ++k) dst[m][k] = *(const LAS f16x8*)(lds + G_SA(b, h) + aoff + m * 2048 + k * 1024); } while (0)
#define G_LDB(dst, b, h) do { _Pragma("unroll") for (int n = 0; n < 2; ++n) _Pragma("unroll") for (int k = 0; k < 2; ++k) dst[n][k] = *(const LAS f16x8*)(lds + G_SB(b, h) + boff + n * 2048 + k * 1024); } while (0)
#define G_MMA(ai, bj, At, Bt) do { __builtin_amdgcn_s_setprio(1); _Pragma("unroll") for (int m = 0; m < 4; ++m) _Pragma("unroll") for (int n = 0; n < 2; ++n) _Pragma("unroll") for (int k = 0; k < 2; ++k) \
        acc[ai][bj][m][n] = __builtin_amdgcn_mfma_f32_16x16x32_f16(Bt[n][k], At[m][k], acc[ai][bj][m][n], 0, 0, 0); __builtin_amdgcn_s_setprio(0); } while (0)
#define G_WAIT_V(n) asm volatile("s_waitcnt vmcnt(" #n ")" ::: "memory")
#define G_WAIT_L(n) asm volatile("s_waitcnt lgkmcnt(" #n ")" ::: "memory")
#define G_BAR __builtin_amdgcn_s_barrier()
#define G_SCHED __builtin_amdgcn_sched_barrier(0)
#define G_NEXT(codevar) do { if (wid == 0) { bool _d = false; const int _c = draw_unit(p, seq, xcc, lane); if (_c >= 0) wait_group(p, seq, _c, U_T0(_c) >> 5, lane, _d); \
            if (lane == 0) { bc[0] = _c; if (_d) bc[1] = 1; } STAMP(0x2000000 + (_c & 0xfffff)); asm volatile("s_waitcnt lgkmcnt(0)" ::: "memory"); } \
        asm volatile("" ::: "memory"); G_BAR; asm volatile("" ::: "memory"); codevar = __builtin_amdgcn_readfirstlane(bc[0]); } while (0)
#define G_GROUP(code, g) do { if (wid == 0) { bool _d = false; STAMP(0x6000000 + (g)); wait_group(p, seq, code, g, lane, _d); STAMP(0x7000000 + (g)); if (lane == 0 && _d) bc[1] = 1; asm volatile("s_waitcnt lgkmcnt(0)" ::: "memory"); } \
        asm volatile("" ::: "memory"); G_BAR; asm volatile("" ::: "memory"); } while (0)

    if (tid == 0) { bc[0] = -1; bc[1] = 0; __builtin_amdgcn_fence(__ATOMIC_ACQUIRE, ""); asm volatile("s_waitcnt vmcnt(0)" ::: "memory"); }
    __syncthreads();
    int cur;
    G_NEXT(cur);
    if (cur < 0) return;
    f32x4 acc[2][2][4][2];
#pragma unroll
    for (int a = 0; a < 2; ++a)
#pragma unroll
        for (int b = 0; b < 2; ++b)
#pragma unroll
            for (int m = 0; m < 4; ++m)
#pragma unroll
                for (int n = 0; n < 2; ++n) acc[a][b][m][n] = (f32x4){0.f, 0.f, 0.f, 0.f};
    f16x8 At[4][2], B0[2][2], B1[2][2];
    const char* cA = Abase + (size_t)(cur & 255) * tstep; const char* cB = Bbase + (size_t)((cur >> 8) & 255) * tstep;
    {
        const size_t k0 = KOFF(U_T0(cur)), k1 = KOFF(U_T0(cur) + 1);
        G_STAGE(G_SB(0, 0), cB + k0); G_STAGE(G_SA(0, 0), cA + k0); G_STAGE(G_SB(0, 1), cB + hstep + k0); G_STAGE(G_SA(0, 1), cA + hstep + k0);
        if (wr == 1) G_BAR;
        G_WAIT_V(4); G_BAR;
        G_STAGE(G_SB(1, 0), cB + k1); G_STAGE(G_SA(1, 0), cA + k1); G_STAGE(G_SB(1, 1), cB + hstep + k1);
        G_WAIT_V(6); G_BAR;
    }
    for (;;) {
        int nxt = -1;
        const int tb = U_T0(cur), te = tb + U_NT(cur);
        for (int t = tb; t < te; t += 2) {
            const bool last = (t == te - 2);
            const char* a2; const char* b2; const char* a3; const char* b3;
            if (last) {
                G_NEXT(nxt);
                const int u = nxt >= 0 ? nxt : cur;
                const char* nA = Abase + (size_t)(u & 255) * tstep; const char* nB = Bbase + (size_t)((u >> 8) & 255) * tstep;
                const size_t k0 = KOFF(U_T0(u)), k1 = KOFF(U_T0(u) + 1);
                a2 = nA + k0; b2 = nB + k0; a3 = nA + k1; b3 = nB + k1;
            } else {
                if (((t + 2) & 31) == 0) G_GROUP(cur, (t + 2) >> 5);
                const size_t k2 = KOFF(t + 2), k3 = KOFF(t + 3);
                a2 = cA + k2; b2 = cB + k2; a3 = cA + k3; b3 = cB + k3;
            }
            const char* a1 = cA + KOFF(t + 1);
            G_LDB(B0, 0, 0); G_SCHED; G_LDA(At, 0, 0); G_STAGE(G_SA(1, 1), a1 + hstep);
            G_WAIT_L(8); G_BAR; G_WAIT_L(0); G_MMA(0, 0, At, B0); G_BAR; G_SCHED;
            G_LDB(B1, 0, 1); G_STAGE(G_SB(0, 0), b2);
            G_BAR; G_WAIT_L(0); G_MMA(0, 1, At, B1); G_BAR;
            G_LDA(At, 0, 1); G_STAGE(G_SA(0, 0), a2);
            G_BAR; G_WAIT_L(0); G_MMA(1, 0, At, B0); G_BAR; G_SCHED;
            G_STAGE(G_SB(0, 1), b2 + hstep);
            G_WAIT_V(6); G_BAR; G_MMA(1, 1, At, B1); G_BAR;
            G_LDB(B0, 1, 0); G_SCHED; G_LDA(At, 1, 0); G_STAGE(G_SA(0, 1), a2 + hstep);
            G_WAIT_L(8); G_BAR; G_WAIT_L(0); G_MMA(0, 0, At, B0); G_BAR; G_SCHED;
            G_LDB(B1, 1, 1); G_STAGE(G_SB(1, 0), b3);
            G_BAR; G_WAIT_L(0); G_MMA(0, 1, At, B1); G_BAR;
            G_LDA(At, 1, 1); G_STAGE(G_SA(1, 0), a3);
            G_BAR; G_WAIT_L(0); G_MMA(1, 0, At, B0); G_BAR; G_SCHED;
            G_STAGE(G_SB(1, 1), b3 + hstep);
            G_WAIT_V(6); G_BAR; G_MMA(1, 1, At, B1); G_BAR;
        }
        {
            dead = (bc[1] != 0);
            bool fin = true;
            if (cur & (1 << 18)) {
                const int slot = cur >> 20, kq = (cur >> 16) & 3;
                char* pbase = p.ws[p.rank] + OFF_PART + (size_t)slot * NG * TILE_BYTES + (size_t)tid * 16;
                f32x4* mine = (f32x4*)(pbase + (size_t)kq * TILE_BYTES);
#pragma unroll
                for (int i = 0; i < 32; ++i)
                    asm volatile("global_store_dwordx4 %0, %1, off sc1\n\ts_nop 1" :: "v"(mine + i * 512), "v"(acc[i >> 4][(i >> 3) & 1][(i >> 1) & 3][i & 1]) : "memory");
                asm volatile("s_waitcnt vmcnt(0)" ::: "memory");
                unsigned tk = 0;
                if (lane == 0) tk = __hip_atomic_fetch_add((unsigned*)(p.ws[p.rank] + OFF_CNT + (size_t)(slot * 8 + wid) * 64), 1u, __ATOMIC_RELAXED, __HIP_MEMORY_SCOPE_AGENT);
                tk = __builtin_amdgcn_readfirstlane(tk);
                fin = ((tk & (NG - 1)) == NG - 1);
                if (fin) {
                    __builtin_amdgcn_fence(__ATOMIC_ACQUIRE, "agent");
                    asm volatile("s_waitcnt vmcnt(0)" ::: "memory");
                    const f32x4* o1 = (const f32x4*)(pbase + (size_t)((kq + 1) & (NG - 1)) * TILE_BYTES);
                    const f32x4* o2 = (const f32x4*)(pbase + (size_t)((kq + 2) & (NG - 1)) * TILE_BYTES);
                    const f32x4* o3 = (const f32x4*)(pbase + (size_t)((kq + 3) & (NG - 1)) * TILE_BYTES);
#pragma unroll
                    for (int i0 = 0; i0 < 32; i0 += 8) {
                        f32x4 t1[8], t2[8], t3[8];
#pragma unroll
                        for (int i = 0; i < 8; ++i) { t1[i] = o1[(i0 + i) * 512]; t2[i] = o2[(i0 + i) * 512]; t3[i] = o3[(i0 + i) * 512]; }
#pragma unroll
                        for (int i = 0; i < 8; ++i) { const int ii = i0 + i; acc[ii >> 4][(ii >> 3) & 1][(ii >> 1) & 3][ii & 1] += (t1[i] + t2[i]) + t3[i]; }
                    }
                    const unsigned tv = __hip_atomic_load((unsigned*)(p.ws[p.rank] + OFF_TMO), __ATOMIC_RELAXED, __HIP_MEMORY_SCOPE_AGENT);
                    if (__builtin_amdgcn_readfirstlane(tv) == seq) dead = true;
                }
            }
            if (cur & (1 << 19)) {
                const float qn = __builtin_nanf("");
                const int row0 = (cur & 255) * BM + wr * 64 + fr, col0 = ((cur >> 8) & 255) * BM + wc * 32 + 4 * fq;
#pragma unroll
                for (int ai = 0; ai < 2; ++ai)
#pragma unroll
                    for (int m = 0; m < 4; ++m) {
                        float* rowp = p.out + (size_t)(row0 + ai * HALF + m * 16) * MN + col0;
#pragma unroll
                        for (int bj = 0; bj < 2; ++bj)
#pragma unroll
                            for (int n = 0; n < 2; ++n) { const f32x4 v = dead ? (f32x4){qn, qn, qn, qn} : acc[ai][bj][m][n];
                                asm volatile("global_store_dwordx4 %0, %1, off sc1\n\ts_nop 1" :: "v"(rowp + bj * HALF + n * 16), "v"(v) : "memory"); }
                    }
                asm volatile("s_waitcnt vmcnt(0)" ::: "memory");
                if (lane == 0) __hip_atomic_fetch_add((unsigned*)(p.ws[p.rank] + OFF_DONE + (size_t)(cur >> 20) * 64), 1u, __ATOMIC_RELAXED, __HIP_MEMORY_SCOPE_AGENT);
            } else if (fin) {
                const float qn = __builtin_nanf("");
                const int row0 = (cur & 255) * BM + wr * 64 + fr, col0 = ((cur >> 8) & 255) * BM + wc * 32 + 4 * fq;
#pragma unroll
                for (int ai = 0; ai < 2; ++ai)
#pragma unroll
                    for (int m = 0; m < 4; ++m) {
                        float* rowp = p.out + (size_t)(row0 + ai * HALF + m * 16) * MN + col0;
#pragma unroll
                        for (int bj = 0; bj < 2; ++bj)
#pragma unroll
                            for (int n = 0; n < 2; ++n) *(f32x4*)(rowp + bj * HALF + n * 16) = dead ? (f32x4){qn, qn, qn, qn} : acc[ai][bj][m][n];
                    }
            }
        }
        STAMP(0x3000000 + (cur & 0xfffff));
        if (nxt < 0) break;
#pragma unroll
        for (int a = 0; a < 2; ++a)
#pragma unroll
            for (int b = 0; b < 2; ++b)
#pragma unroll
                for (int m = 0; m < 4; ++m)
#pragma unroll
                    for (int n = 0; n < 2; ++n) acc[a][b][m][n] = (f32x4){0.f, 0.f, 0.f, 0.f};
        cur = nxt; cA = Abase + (size_t)(cur & 255) * tstep; cB = Bbase + (size_t)((cur >> 8) & 255) * tstep;
    }
    G_WAIT_V(0);
    if (wr == 0) G_BAR;
    G_BAR;
}

__global__ __launch_bounds__(512, 2) void fused_kernel(Params p) {
    extern __shared__ __attribute__((aligned(16))) unsigned char shm[];
    LAS unsigned char* lds = (LAS unsigned char*)shm;
    volatile LAS unsigned* sq = (volatile LAS unsigned*)(lds + STAGE_BYTES);
    if (threadIdx.x == 0) {
        const unsigned t = __hip_atomic_fetch_add((unsigned*)(p.ws[p.rank] + OFF_CALL), 1u, __ATOMIC_RELAXED, __HIP_MEMORY_SCOPE_AGENT);
        sq[0] = t / GRID + 1u;
    }
    __syncthreads();
    const unsigned seq = __builtin_amdgcn_readfirstlane(sq[0]);
    int nst = 0;
    STAMP(1);
    if ((int)blockIdx.x < NPUSH) { push_phase(p, (int)blockIdx.x, seq, nst); forward_phase(p, lds, (int)blockIdx.x, seq, nst); }
    else gemm_phase(p, lds, seq, nst);
}
}

extern "C" void kernel_launch(int rank, int nranks, ncclComm_t comm,
                              hipStream_t stream,
                              void* const* d_in, const int* in_sizes, int n_in,
                              void* d_out, int out_size,
                              const int* in_dtypes, int out_dtype,
                              void* const* d_out_all,
                              void* const* d_ws_all, size_t ws_bytes) {
    if (nranks != NR || ws_bytes < WS_NEED || n_in < 2) { fprintf(stderr, "unsupported configuration\n"); return; }
    static bool inits[NR] = {};
    static Params Ps[NR];
    if (rank < 0 || rank >= NR) return;
    bool& init = inits[rank];
    Params& P = Ps[rank];
    if (!init) {
        HIP_CHECK(hipFuncSetAttribute((const void*)fused_kernel, hipFuncAttributeMaxDynamicSharedMemorySize, LDS_BYTES));
        std::vector<int> own;
        for (int h = 0; h < 2; ++h) for (int pm = SHBASE; pm < NPAN; ++pm) if (sh_owner(pm, h * 16) == rank) for (int pn = h * 16; pn < h * 16 + 16; ++pn) own.push_back(pm | pn << 5);
        for (int i = 0; i < NOWN; ++i) P.own[i] = (unsigned short)own[i];
        for (int x = 0; x < NR; ++x) {
            std::vector<std::pair<int, int>> v;
            for (int i = 0; i < SHBASE / 2; ++i) for (int j = 0; j < 8; ++j) {
                const int pm = 2 * i + (x & 1), pn = 4 * j + (x >> 1);
                const int region = (pm < 16 ? 0 : 2) + (pn < 16 ? 0 : 1);
                v.push_back({region * 4096 + 1024 + j * 64 + i, pm | pn << 5});
            }
            for (int i = x; i < NOWN; i += NR) { const int pn = (own[i] >> 5) & 31; v.push_back({(pn < 16 ? 0 : 1) * 4096 + i, own[i] | (i + 1) << 10}); }
            std::sort(v.begin(), v.end());
            if ((int)v.size() != QTILES) { fprintf(stderr, "list size mismatch\n"); return; }
            for (int k = 0; k < QTILES; ++k) P.order[x][k] = (unsigned short)v[k].second;
        }
        init = true;
    }
    P.x = (const float*)d_in[0]; P.w = (const float*)d_in[1]; P.out = (float*)d_out; P.rank = rank;
    for (int r = 0; r < NR; ++r) { P.ws[r] = (char*)d_ws_all[r]; P.outs[r] = (float*)d_out_all[r]; }
    hipLaunchKernelGGL(fused_kernel, dim3(GRID), dim3(512), LDS_BYTES, stream, P);
    HIP_CHECK(hipGetLastError());
}
```

```cpp
#include <hip/hip_runtime.h>
#include <rccl/rccl.h>
#include <cstdio>
#include <cstdint>
#include <algorithm>
#include <vector>

#define HIP_CHECK(expr) do { hipError_t _e = (expr); if (_e != hipSuccess) { \
    fprintf(stderr, "HIP error %s at %s:%d\n", hipGetErrorString(_e), __FILE__, __LINE__); \
    return; } } while (0)

namespace {
#define LAS __attribute__((address_space(3)))
typedef _Float16 f16x8 __attribute__((ext_vector_type(8)));
typedef float f32x4 __attribute__((ext_vector_type(4)));
typedef float f32x2 __attribute__((ext_vector_type(2)));
typedef unsigned u32x4 __attribute__((ext_vector_type(4)));

constexpr int NR = 8;
constexpr int MN = 8192;
constexpr int KSH = 1024;
constexpr int BM = 256, BK = 64, HALF = 128;
constexpr int HTB = HALF * BK * 2;
constexpr int NKT = NR * KSH / BK;
constexpr int KT_PER_RANK = KSH / BK;
constexpr int NPAN = MN / BM;
constexpr int NG = 4;
constexpr size_t MAT_BYTES = (size_t)MN * NR * KSH * 2;
constexpr size_t CTRL_BYTES = (size_t)1 << 20;
#define NSPLIT 12
#define NSH 12
static_assert(NSH % 4 == 0 && NSH <= 12, "NSH");
constexpr int SHBASE = NPAN - NSH;
constexpr int NOWN = NSH * NPAN / NR;
__host__ __device__ constexpr int sh_owner(int pm, int pn) { const int u = (pm - SHBASE) * 2 + (pn >> 4); return (u + (u >> 3)) & (NR - 1); }
constexpr int QTILES = (SHBASE * NPAN + NOWN) / NR;
#define A0 22
#define S0 12
constexpr int NMID = QTILES - A0 - S0 - NSPLIT;
constexpr int QLEN = A0 + S0 * NG + NMID + NSPLIT * NG;
constexpr int NSLOT = S0 + NSPLIT;
constexpr size_t TILE_BYTES = (size_t)BM * BM * 4;
constexpr size_t OFF_PART = CTRL_BYTES + 4 * MAT_BYTES;
constexpr size_t WS_NEED = OFF_PART + (size_t)NR * (S0 + NSPLIT) * NG * TILE_BYTES;
constexpr int STAGE_BYTES = 8 * HTB;
constexpr int LDS_BYTES = STAGE_BYTES + 64;
constexpr int GRID = 256;
#define NPUSH 32
#define NFWD 0
constexpr unsigned long long DEADLINE = 50000000ull;

constexpr size_t OFF_CALL = 0;
constexpr size_t OFF_TMO = 128;
constexpr size_t OFF_QUEUE = 1024;
constexpr size_t OFF_CENSUS = 4096;
#define FQ 2
constexpr size_t OFF_FWDQ = 3072;
constexpr size_t OFF_AMAX = 5120;
constexpr size_t OFF_SCALE = 6144;
constexpr size_t OFF_FLAG = 8192;
constexpr size_t OFF_GRP = 64 * 1024;
constexpr size_t OFF_CNT = 256 * 1024;
constexpr size_t OFF_DONE = 384 * 1024;
constexpr size_t OFF_OUTF = 448 * 1024;
__host__ __device__ constexpr size_t flag_off(int src, int reg, int g) { return OFF_FLAG + (size_t)((src * 3 + reg) * NG + g) * 128; }
constexpr unsigned FLAG_PER_CALL = NPUSH;

#define STAMP(tag) do { } while (0)
#define REL_SYS() __builtin_amdgcn_fence(__ATOMIC_RELEASE, "")
#define STMODE 1
#define ST_REMOTE(ptr, val) asm volatile("global_store_dwordx4 %0, %1, off sc0 sc1" :: "v"(ptr), "v"(val) : "memory")
struct Params {
    const float* x; const float* w; float* out;
    char* ws[NR];
    float* outs[NR];
    int rank;
    unsigned short order[NR][128];
    unsigned short own[64];
};

__device__ __forceinline__ int lds_byte(int r, int c) { const int st = (r >> 4) * 2 + (c >> 5), rr = r & 15, cc = c & 31, ob = rr * 64 + cc * 2; return st * 1024 + (ob ^ (((ob >> 9) & 1) << 5)); }
__device__ __forceinline__ void stage_rc(int b, int& R, int& C) { const int st = b / 1024, sb = b % 1024, swz = sb ^ (((sb >> 9) & 1) << 5); R = (st >> 1) * 16 + swz / 64; C = (st & 1) * 32 + (swz % 64) / 2; }

__device__ __forceinline__ unsigned pack2(float a, float b, float sc) {
    const _Float16 ha = (_Float16)(a * sc), hb = (_Float16)(b * sc);
    return (unsigned)__builtin_bit_cast(unsigned short, ha) | ((unsigned)__builtin_bit_cast(unsigned short, hb) << 16);
}

__device__ __forceinline__ unsigned seq_inc(unsigned long long* q, unsigned seq, unsigned add) {
    unsigned long long old = __hip_atomic_load(q, __ATOMIC_RELAXED, __HIP_MEMORY_SCOPE_AGENT);
    for (;;) {
        const unsigned cnt = ((unsigned)(old >> 32) == seq) ? (unsigned)old : 0u;
        if (__hip_atomic_compare_exchange_strong(q, &old, ((unsigned long long)seq << 32) | (cnt + add), __ATOMIC_RELAXED, __ATOMIC_RELAXED, __HIP_MEMORY_SCOPE_AGENT)) return cnt;
    }
}
__device__ __forceinline__ unsigned seq_get(unsigned long long* q, unsigned seq) {
    const unsigned long long v = __hip_atomic_load(q, __ATOMIC_RELAXED, __HIP_MEMORY_SCOPE_AGENT);
    return ((unsigned)(v >> 32) == seq) ? (unsigned)v : 0u;
}

__device__ __forceinline__ void push_phase(const Params& p, int pw, unsigned seq, float sx, float sw, int& nst) {
    const int tid = threadIdx.x;
    const size_t par = (size_t)(seq & 1) * 2 * MAT_BYTES;
    int R[2], C[2];
    stage_rc(tid * 16, R[0], C[0]); stage_rc(tid * 16 + 8192, R[1], C[1]);
    const int wkg = tid & 7, wng = tid >> 3;
    char* dst[NR];
#pragma unroll
    for (int q = 0; q < NR; ++q) dst[q] = p.ws[(p.rank + q) & (NR - 1)];
    char* myws = p.ws[p.rank];
    const unsigned xcc = (unsigned)__builtin_amdgcn_s_getreg((3 << 11) | 20) & 7u;
    unsigned ngrp = 0;
    if (tid == 0) {
        seq_inc((unsigned long long*)(myws + OFF_CENSUS + xcc * 64), seq, 1u);
        seq_inc((unsigned long long*)(myws + OFF_CENSUS + 512), seq, 1u);
        const unsigned long long t0 = __builtin_amdgcn_s_memrealtime();
        while (seq_get((unsigned long long*)(myws + OFF_CENSUS + 512), seq) < (unsigned)NPUSH) {
            if (__builtin_amdgcn_s_memrealtime() - t0 > DEADLINE) { __hip_atomic_store((unsigned*)(myws + OFF_TMO), seq, __ATOMIC_RELAXED, __HIP_MEMORY_SCOPE_AGENT); break; }
            __builtin_amdgcn_s_sleep(1);
        }
        ngrp = seq_get((unsigned long long*)(myws + OFF_CENSUS + xcc * 64), seq);
    }
    for (int step = 0; step < 2 * NG + 1; ++step) {
        const int reg = step >> 2, g = step & 3;
        const int nitem = reg == 0 ? (2 * 16 + NSH) * 8 : reg == 1 ? 16 * 8 : (SHBASE - 16) * 32;
        for (int it0 = pw; it0 < nitem; it0 += 2 * NPUSH) {
            u32x4 o[2][2]; size_t off[2][2]; unsigned mask[2];
            f32x4 a[2][2][2]; f32x2 bq[2][8]; int mats[2];
#pragma unroll
            for (int z = 0; z < 2; ++z) {
                const int it = it0 + z * NPUSH;
                mask[z] = it < nitem ? 0xffu : 0u;
                const int itc = it < nitem ? it : it0;
                const int hb = itc & 1, i2 = itc >> 1;
                const int pi = reg == 2 ? i2 >> 4 : i2 >> 2, kt = reg == 2 ? (i2 & 15) : g * 4 + (i2 & 3);
                int mat, j;
                if (reg == 0) { if (pi < 16) { mat = 0; j = pi; } else if (pi < 32) { mat = 1; j = pi - 16; }
                                else { mat = 0; j = SHBASE + pi - 32; mask[z] &= (1u << sh_owner(j, 0)) | (1u << sh_owner(j, 16)); } }
                else if (reg == 1) { mat = 1; j = 16 + pi; }
                else { mat = 0; j = 16 + pi; }
                mats[z] = mat;
                const size_t ioff = CTRL_BYTES + par + (size_t)mat * MAT_BYTES + ((size_t)(j * 2 + hb) * NKT + p.rank * KT_PER_RANK + kt) * HTB;
                if (mat == 0) {
#pragma unroll
                    for (int v = 0; v < 2; ++v) {
                        const float* sp = p.x + (size_t)(j * BM + hb * HALF + R[v]) * KSH + kt * BK + C[v];
                        a[z][v][0] = *(const f32x4*)sp; a[z][v][1] = *(const f32x4*)(sp + 4);
                        off[z][v] = ioff + v * 8192 + tid * 16;
                    }
                } else {
                    const float* sp = p.w + (size_t)(kt * BK + wkg * 8) * MN + j * BM + hb * HALF + wng * 2;
#pragma unroll
                    for (int e = 0; e < 8; ++e) bq[z][e] = *(const f32x2*)(sp + (size_t)e * MN);
                    off[z][0] = ioff + lds_byte(wng * 2, wkg * 8); off[z][1] = ioff + lds_byte(wng * 2 + 1, wkg * 8);
                }
            }
#pragma unroll
            for (int z = 0; z < 2; ++z) {
                if (mats[z] == 0) {
#pragma unroll
                    for (int v = 0; v < 2; ++v) {
                        o[z][v].x = pack2(a[z][v][0][0], a[z][v][0][1], sx); o[z][v].y = pack2(a[z][v][0][2], a[z][v][0][3], sx);
                        o[z][v].z = pack2(a[z][v][1][0], a[z][v][1][1], sx); o[z][v].w = pack2(a[z][v][1][2], a[z][v][1][3], sx);
                    }
                } else {
#pragma unroll
                    for (int e = 0; e < 2; ++e) {
                        o[z][e].x = pack2(bq[z][0][e], bq[z][1][e], sw); o[z][e].y = pack2(bq[z][2][e], bq[z][3][e], sw);
                        o[z][e].z = pack2(bq[z][4][e], bq[z][5][e], sw); o[z][e].w = pack2(bq[z][6][e], bq[z][7][e], sw);
                    }
                }
#pragma unroll
                for (int q = 0; q < NR; ++q) {
                    if (!((mask[z] >> ((p.rank + q) & (NR - 1))) & 1u)) continue;
#pragma unroll
                    for (int v = 0; v < 2; ++v) ST_REMOTE(dst[q] + off[z][v], o[z][v]);
                }
            }
        }
        asm volatile("s_waitcnt vmcnt(0)" ::: "memory");
        __builtin_amdgcn_s_barrier();
        if (tid == 0) {
            const unsigned old = seq_inc((unsigned long long*)(myws + OFF_GRP + (size_t)(xcc * 16 + step) * 64), seq, 1u);
            if (old + 1u == ngrp) {
                REL_SYS();
                asm volatile("s_waitcnt vmcnt(0)" ::: "memory");
                for (int gg = (reg == 2 ? 0 : g); gg <= (reg == 2 ? NG - 1 : g); ++gg)
#pragma unroll
                    for (int q = 0; q < NR; ++q)
                        __hip_atomic_fetch_add((unsigned*)(dst[q] + flag_off(p.rank, reg, gg)), ngrp, __ATOMIC_RELAXED, __HIP_MEMORY_SCOPE_SYSTEM);
            }
        }
        STAMP(0x1000000 + step);
    }
}

__device__ __forceinline__ void forward_phase(const Params& p, LAS unsigned char* lds, int pw, unsigned seq, int& nst) {
    const int tid = threadIdx.x;
    char* myws = p.ws[p.rank];
    unsigned* tmo = (unsigned*)(myws + OFF_TMO);
    volatile LAS int* fb = (volatile LAS int*)(lds + 64);
    for (;;) {
        __syncthreads();
        if (tid == 0) fb[0] = (int)seq_inc((unsigned long long*)(myws + OFF_FWDQ), seq, 1u);
        __syncthreads();
        const int iu = fb[0];
        if (iu >= NOWN * FQ) break;
        const int i = iu / FQ, qr = iu % FQ;
        if (tid == 0) {
            const unsigned* d = (const unsigned*)(myws + OFF_DONE + (size_t)i * 64);
            const unsigned long long t0 = __builtin_amdgcn_s_memrealtime();
            while ((int)(__hip_atomic_load(d, __ATOMIC_RELAXED, __HIP_MEMORY_SCOPE_AGENT) - seq * 8u) < 0) {
                if (__hip_atomic_load(tmo, __ATOMIC_RELAXED, __HIP_MEMORY_SCOPE_AGENT) == seq || __builtin_amdgcn_s_memrealtime() - t0 > DEADLINE) {
                    __hip_atomic_store(tmo, seq, __ATOMIC_RELAXED, __HIP_MEMORY_SCOPE_AGENT); break; }
                __builtin_amdgcn_s_sleep(16);
            }
            __builtin_amdgcn_fence(__ATOMIC_ACQUIRE, "agent");
            asm volatile("s_waitcnt vmcnt(0)" ::: "memory");
        }
        __syncthreads();
        const int t = p.own[i];
        const size_t base = ((size_t)(t & 31) * BM) * MN + (size_t)((t >> 5) & 31) * BM;
        for (int r0 = qr * (BM / FQ); r0 < (qr + 1) * (BM / FQ); r0 += 32) {
            f32x4 v[4]; size_t off[4];
#pragma unroll
            for (int u = 0; u < 4; ++u) { off[u] = base + (size_t)(r0 + u * 8 + (tid >> 6)) * MN + (tid & 63) * 4; v[u] = *(const f32x4*)(p.out + off[u]); }
#pragma unroll
            for (int q = 1; q < NR; ++q) {
                float* o = p.outs[(p.rank + q) & (NR - 1)];
#pragma unroll
                for (int u = 0; u < 4; ++u) ST_REMOTE(o + off[u], v[u]);
            }
        }
        asm volatile("s_waitcnt vmcnt(0)" ::: "memory");
        __syncthreads();
        if (tid == 0) {
            REL_SYS();
            asm volatile("s_waitcnt vmcnt(0)" ::: "memory");
#pragma unroll
            for (int q = 1; q < NR; ++q)
                __hip_atomic_fetch_add((unsigned*)(p.ws[(p.rank + q) & (NR - 1)] + OFF_OUTF + (size_t)p.rank * 128), 1u, __ATOMIC_RELAXED, __HIP_MEMORY_SCOPE_SYSTEM);
        }
        STAMP(0x4000000 + iu);
    }
    if (pw == 0 && tid < 64) {
        const int src = tid & 7;
        const unsigned* f = (const unsigned*)(myws + OFF_OUTF + (size_t)src * 128);
        const unsigned want = seq * (unsigned)(NOWN * FQ);
        const unsigned long long t0 = __builtin_amdgcn_s_memrealtime();
        bool dead = false;
        for (;;) {
            const unsigned v = __hip_atomic_load(f, __ATOMIC_RELAXED, __HIP_MEMORY_SCOPE_SYSTEM);
            if (__all(src == p.rank || (int)(v - want) >= 0)) break;
            const unsigned tv = __hip_atomic_load(tmo, __ATOMIC_RELAXED, __HIP_MEMORY_SCOPE_AGENT);
            if (__builtin_amdgcn_readfirstlane(tv) == seq || __builtin_amdgcn_s_memrealtime() - t0 > DEADLINE) { dead = true; break; }
            __builtin_amdgcn_s_sleep(16);
        }
        __builtin_amdgcn_fence(__ATOMIC_ACQUIRE, "");
        if (dead && tid == 0) { __hip_atomic_store(tmo, seq, __ATOMIC_RELAXED, __HIP_MEMORY_SCOPE_AGENT); p.out[0] = __builtin_nanf(""); }
        STAMP(0x5000000);
    }
}

__device__ __forceinline__ void wait_group(const Params& p, unsigned seq, int code, int g, int lane, bool& dead) {
    char* myws = p.ws[p.rank];
    const int pm = code & 255, pn = (code >> 8) & 255;
    const int reg = (lane & 1) ? (pn < NPAN / 2 ? 0 : 1) : ((pm < NPAN / 2 || pm >= SHBASE) ? 0 : 2);
    const unsigned* f = (const unsigned*)(myws + flag_off((lane >> 1) & 7, reg, g));
    const unsigned want = seq * FLAG_PER_CALL;
    unsigned* tmo = (unsigned*)(myws + OFF_TMO);
    const unsigned long long t0 = __builtin_amdgcn_s_memrealtime();
    for (;;) {
        const unsigned v = __hip_atomic_load(f, __ATOMIC_RELAXED, __HIP_MEMORY_SCOPE_SYSTEM);
        if (__all((int)(v - want) >= 0)) break;
        const unsigned tv = __hip_atomic_load(tmo, __ATOMIC_RELAXED, __HIP_MEMORY_SCOPE_AGENT);
        if (__builtin_amdgcn_readfirstlane(tv) == seq || __builtin_amdgcn_s_memrealtime() - t0 > DEADLINE) {
            if (lane == 0) __hip_atomic_store(tmo, seq, __ATOMIC_RELAXED, __HIP_MEMORY_SCOPE_AGENT);
            dead = true; break;
        }
        __builtin_amdgcn_s_sleep(16);
    }
    __builtin_amdgcn_fence(__ATOMIC_ACQUIRE, "wavefront");
    asm volatile("s_waitcnt vmcnt(0)" ::: "memory");
}

__device__ __forceinline__ int draw_unit(const Params& p, unsigned seq, unsigned xcc, int lane) {
    char* myws = p.ws[p.rank];
    int code = -1;
    if (lane == 0) {
        for (int s = 0; s < NR && code < 0; ++s) {
            const unsigned q = (xcc + s) & (NR - 1);
            unsigned long long* qp = (unsigned long long*)(myws + OFF_QUEUE + q * 128);
            unsigned long long old = __hip_atomic_load(qp, __ATOMIC_RELAXED, __HIP_MEMORY_SCOPE_AGENT);
            for (;;) {
                const unsigned cnt = ((unsigned)(old >> 32) == seq) ? (unsigned)old : 0u;
                if (cnt >= (unsigned)QLEN) break;
                const unsigned long long nw = ((unsigned long long)seq << 32) | (cnt + 1u);
                if (__hip_atomic_compare_exchange_strong(qp, &old, nw, __ATOMIC_RELAXED, __ATOMIC_RELAXED, __HIP_MEMORY_SCOPE_AGENT)) {
                    int c = (int)cnt, ti, kq = -1, slot = 0;
                    if (c < A0) ti = c;
                    else if ((c -= A0) < S0 * NG) { ti = A0 + c % S0; kq = c / S0; slot = c % S0; }
                    else if ((c -= S0 * NG) < NMID) ti = A0 + S0 + c;
                    else { c -= NMID; ti = A0 + S0 + NMID + c % NSPLIT; kq = c / NSPLIT; slot = S0 + c % NSPLIT; }
                    const int t = p.order[q][ti];
                    code = (t & 31) | (((t >> 5) & 31) << 8);
                    if (kq >= 0) code |= (kq << 16) | (1 << 18) | (((int)q * NSLOT + slot) << 20);
                    else if (t >> 10) code |= (1 << 19) | (((t >> 10) - 1) << 20);
                    break;
                }
            }
        }
    }
    return __builtin_amdgcn_readfirstlane(code);
}

__device__ __forceinline__ void gemm_phase(const Params& p, LAS unsigned char* lds, unsigned seq, float ox, float ow, int& nst) {
    const int tid = threadIdx.x, wid = __builtin_amdgcn_readfirstlane(tid >> 6), lane = tid & 63, wr = wid >> 2, wc = wid & 3, fr = lane & 15, fq = lane >> 4;
    const unsigned xcc = (unsigned)__builtin_amdgcn_s_getreg((3 << 11) | 20) & 7u;
    volatile LAS int* bc = (volatile LAS int*)(lds + STAGE_BYTES + 16);
    const char* Abase = p.ws[p.rank] + CTRL_BYTES + (size_t)(seq & 1) * 2 * MAT_BYTES;
    const char* Bbase = Abase + MAT_BYTES;
    constexpr size_t hstep = (size_t)NKT * HTB;
    constexpr size_t tstep = 2 * hstep;
    const unsigned voff = (unsigned)tid * 16u;
    const unsigned ldsw = (unsigned)wid * 1024u;
    const int aoff = lds_byte(wr * 64 + fr, fq * 8), boff = lds_byte(wc * 32 + fr, fq * 8);
    const int rank = p.rank;
    bool dead = false;
#define KOFF(T) ((size_t)((((rank + (((T) >> 2) & 7)) & 7) << 4) | (((T) >> 5) << 2) | ((T) & 3)) * HTB)
#define U_T0(c) ((((c) >> 18) & 1) * (((c) >> 16) & 3) * 32)
#define U_NT(c) ((((c) >> 18) & 1) ? 32 : NKT)
#define G_SA(b, h) (((b) * 2 + (h)) * HTB)
#define G_SB(b, h) ((4 + (b) * 2 + (h)) * HTB)
#define G_STAGE(bufoff, gbase) do { _Pragma("unroll") for (int _i = 0; _i < 2; ++_i) \
        __builtin_amdgcn_global_load_lds((const unsigned*)((const char*)(gbase) + voff + _i * 8192), (LAS unsigned*)(lds + (bufoff) + ldsw + _i * 8192), 16, 0, 0); } while (0)
#define G_LDA(dst, b, h) do { _Pragma("unroll") for (int m = 0; m < 4; ++m) _Pragma("unroll") for (int k = 0; k < 2; ++k) dst[m][k] = *(const LAS f16x8*)(lds + G_SA(b, h) + aoff + m * 2048 + k * 1024); } while (0)
#define G_LDB(dst, b, h) do { _Pragma("unroll") for (int n = 0; n < 2; ++n) _Pragma("unroll") for (int k = 0; k < 2; ++k) dst[n][k] = *(const LAS f16x8*)(lds + G_SB(b, h) + boff + n * 2048 + k * 1024); } while (0)
#define G_MMA(ai, bj, At, Bt) do { __builtin_amdgcn_s_setprio(1); _Pragma("unroll") for (int m = 0; m < 4; ++m) _Pragma("unroll") for (int n = 0; n < 2; ++n) _Pragma("unroll") for (int k = 0; k < 2; ++k) \
        acc[ai][bj][m][n] = __builtin_amdgcn_mfma_f32_16x16x32_f16(Bt[n][k], At[m][k], acc[ai][bj][m][n], 0, 0, 0); __builtin_amdgcn_s_setprio(0); } while (0)
#define G_WAIT_V(n) asm volatile("s_waitcnt vmcnt(" #n ")" ::: "memory")
#define G_WAIT_L(n) asm volatile("s_waitcnt lgkmcnt(" #n ")" ::: "memory")
#define G_BAR __builtin_amdgcn_s_barrier()
#define G_SCHED __builtin_amdgcn_sched_barrier(0)
#define G_NEXT(codevar) do { if (wid == 0) { bool _d = false; const int _c = draw_unit(p, seq, xcc, lane); if (_c >= 0) wait_group(p, seq, _c, U_T0(_c) >> 5, lane, _d); \
            if (lane == 0) { bc[0] = _c; if (_d) bc[1] = 1; } STAMP(0x2000000 + (_c & 0xfffff)); asm volatile("s_waitcnt lgkmcnt(0)" ::: "memory"); } \
        asm volatile("" ::: "memory"); G_BAR; asm volatile("" ::: "memory"); codevar = __builtin_amdgcn_readfirstlane(bc[0]); } while (0)
#define G_GROUP(code, g) do { if (wid == 0) { bool _d = false; STAMP(0x6000000 + (g)); wait_group(p, seq, code, g, lane, _d); STAMP(0x7000000 + (g)); if (lane == 0 && _d) bc[1] = 1; asm volatile("s_waitcnt lgkmcnt(0)" ::: "memory"); } \
        asm volatile("" ::: "memory"); G_BAR; asm volatile("" ::: "memory"); } while (0)

    if (tid == 0) { bc[0] = -1; bc[1] = 0; __builtin_amdgcn_fence(__ATOMIC_ACQUIRE, ""); asm volatile("s_waitcnt vmcnt(0)" ::: "memory"); }
    __syncthreads();
    int cur;
    G_NEXT(cur);
    if (cur < 0) return;
    f32x4 acc[2][2][4][2];
#pragma unroll
    for (int a = 0; a < 2; ++a)
#pragma unroll
        for (int b = 0; b < 2; ++b)
#pragma unroll
            for (int m = 0; m < 4; ++m)
#pragma unroll
                for (int n = 0; n < 2; ++n) acc[a][b][m][n] = (f32x4){0.f, 0.f, 0.f, 0.f};
    f16x8 At[4][2], B0[2][2], B1[2][2];
    const char* cA = Abase + (size_t)(cur & 255) * tstep; const char* cB = Bbase + (size_t)((cur >> 8) & 255) * tstep;
    {
        const size_t k0 = KOFF(U_T0(cur)), k1 = KOFF(U_T0(cur) + 1);
        G_STAGE(G_SB(0, 0), cB + k0); G_STAGE(G_SA(0, 0), cA + k0); G_STAGE(G_SB(0, 1), cB + hstep + k0); G_STAGE(G_SA(0, 1), cA + hstep + k0);
        if (wr == 1) G_BAR;
        G_WAIT_V(4); G_BAR;
        G_STAGE(G_SB(1, 0), cB + k1); G_STAGE(G_SA(1, 0), cA + k1); G_STAGE(G_SB(1, 1), cB + hstep + k1);
        G_WAIT_V(6); G_BAR;
    }
    for (;;) {
        int nxt = -1;
        const int tb = U_T0(cur), te = tb + U_NT(cur);
        for (int t = tb; t < te; t += 2) {
            const bool last = (t == te - 2);
            const char* a2; const char* b2; const char* a3; const char* b3;
            if (last) {
                G_NEXT(nxt);
                const int u = nxt >= 0 ? nxt : cur;
                const char* nA = Abase + (size_t)(u & 255) * tstep; const char* nB = Bbase + (size_t)((u >> 8) & 255) * tstep;
                const size_t k0 = KOFF(U_T0(u)), k1 = KOFF(U_T0(u) + 1);
                a2 = nA + k0; b2 = nB + k0; a3 = nA + k1; b3 = nB + k1;
            } else {
                if (((t + 2) & 31) == 0) G_GROUP(cur, (t + 2) >> 5);
                const size_t k2 = KOFF(t + 2), k3 = KOFF(t + 3);
                a2 = cA + k2; b2 = cB + k2; a3 = cA + k3; b3 = cB + k3;
            }
            const char* a1 = cA + KOFF(t + 1);
            G_LDB(B0, 0, 0); G_SCHED; G_LDA(At, 0, 0); G_STAGE(G_SA(1, 1), a1 + hstep);
            G_WAIT_L(8); G_BAR; G_WAIT_L(0); G_MMA(0, 0, At, B0); G_BAR; G_SCHED;
            G_LDB(B1, 0, 1); G_STAGE(G_SB(0, 0), b2);
            G_BAR; G_WAIT_L(0); G_MMA(0, 1, At, B1); G_BAR;
            G_LDA(At, 0, 1); G_STAGE(G_SA(0, 0), a2);
            G_BAR; G_WAIT_L(0); G_MMA(1, 0, At, B0); G_BAR; G_SCHED;
            G_STAGE(G_SB(0, 1), b2 + hstep);
            G_WAIT_V(6); G_BAR; G_MMA(1, 1, At, B1); G_BAR;
            G_LDB(B0, 1, 0); G_SCHED; G_LDA(At, 1, 0); G_STAGE(G_SA(0, 1), a2 + hstep);
            G_WAIT_L(8); G_BAR; G_WAIT_L(0); G_MMA(0, 0, At, B0); G_BAR; G_SCHED;
            G_LDB(B1, 1, 1); G_STAGE(G_SB(1, 0), b3);
            G_BAR; G_WAIT_L(0); G_MMA(0, 1, At, B1); G_BAR;
            G_LDA(At, 1, 1); G_STAGE(G_SA(1, 0), a3);
            G_BAR; G_WAIT_L(0); G_MMA(1, 0, At, B0); G_BAR; G_SCHED;
            G_STAGE(G_SB(1, 1), b3 + hstep);
            G_WAIT_V(6); G_BAR; G_MMA(1, 1, At, B1); G_BAR;
        }
        {
            dead = (bc[1] != 0);
            bool fin = true;
            if (cur & (1 << 18)) {
                const int slot = cur >> 20, kq = (cur >> 16) & 3;
                char* pbase = p.ws[p.rank] + OFF_PART + (size_t)slot * NG * TILE_BYTES + (size_t)tid * 16;
                f32x4* mine = (f32x4*)(pbase + (size_t)kq * TILE_BYTES);
#pragma unroll
                for (int i = 0; i < 32; ++i)
                    asm volatile("global_store_dwordx4 %0, %1, off sc1\n\ts_nop 1" :: "v"(mine + i * 512), "v"(acc[i >> 4][(i >> 3) & 1][(i >> 1) & 3][i & 1]) : "memory");
                asm volatile("s_waitcnt vmcnt(0)" ::: "memory");
                unsigned tk = 0;
                if (lane == 0) tk = __hip_atomic_fetch_add((unsigned*)(p.ws[p.rank] + OFF_CNT + (size_t)(slot * 8 + wid) * 64), 1u, __ATOMIC_RELAXED, __HIP_MEMORY_SCOPE_AGENT);
                tk = __builtin_amdgcn_readfirstlane(tk);
                fin = ((tk & (NG - 1)) == NG - 1);
                if (fin) {
                    __builtin_amdgcn_fence(__ATOMIC_ACQUIRE, "agent");
                    asm volatile("s_waitcnt vmcnt(0)" ::: "memory");
                    const f32x4* o1 = (const f32x4*)(pbase + (size_t)((kq + 1) & (NG - 1)) * TILE_BYTES);
                    const f32x4* o2 = (const f32x4*)(pbase + (size_t)((kq + 2) & (NG - 1)) * TILE_BYTES);
                    const f32x4* o3 = (const f32x4*)(pbase + (size_t)((kq + 3) & (NG - 1)) * TILE_BYTES);
#pragma unroll
                    for (int i0 = 0; i0 < 32; i0 += 8) {
                        f32x4 t1[8], t2[8], t3[8];
#pragma unroll
                        for (int i = 0; i < 8; ++i) { t1[i] = o1[(i0 + i) * 512]; t2[i] = o2[(i0 + i) * 512]; t3[i] = o3[(i0 + i) * 512]; }
#pragma unroll
                        for (int i = 0; i < 8; ++i) { const int ii = i0 + i; acc[ii >> 4][(ii >> 3) & 1][(ii >> 1) & 3][ii & 1] += (t1[i] + t2[i]) + t3[i]; }
                    }
                    const unsigned tv = __hip_atomic_load((unsigned*)(p.ws[p.rank] + OFF_TMO), __ATOMIC_RELAXED, __HIP_MEMORY_SCOPE_AGENT);
                    if (__builtin_amdgcn_readfirstlane(tv) == seq) dead = true;
                }
            }
            if (cur & (1 << 19)) {
                const float qn = __builtin_nanf("");
                const int row0 = (cur & 255) * BM + wr * 64 + fr, col0 = ((cur >> 8) & 255) * BM + wc * 32 + 4 * fq;
#pragma unroll
                for (int ai = 0; ai < 2; ++ai)
#pragma unroll
                    for (int m = 0; m < 4; ++m) {
                        float* rowp = p.out + (size_t)(row0 + ai * HALF + m * 16) * MN + col0;
#pragma unroll
                        for (int bj = 0; bj < 2; ++bj)
#pragma unroll
                            for (int n = 0; n < 2; ++n) { const f32x4 v = dead ? (f32x4){qn, qn, qn, qn} : (acc[ai][bj][m][n] * ox) * ow;
                                asm volatile("global_store_dwordx4 %0, %1, off sc1\n\ts_nop 1" :: "v"(rowp + bj * HALF + n * 16), "v"(v) : "memory"); }
                    }
                asm volatile("s_waitcnt vmcnt(0)" ::: "memory");
                if (lane == 0) __hip_atomic_fetch_add((unsigned*)(p.ws[p.rank] + OFF_DONE + (size_t)(cur >> 20) * 64), 1u, __ATOMIC_RELAXED, __HIP_MEMORY_SCOPE_AGENT);
            } else if (fin) {
                const float qn = __builtin_nanf("");
                const int row0 = (cur & 255) * BM + wr * 64 + fr, col0 = ((cur >> 8) & 255) * BM + wc * 32 + 4 * fq;
#pragma unroll
                for (int ai = 0; ai < 2; ++ai)
#pragma unroll
                    for (int m = 0; m < 4; ++m) {
                        float* rowp = p.out + (size_t)(row0 + ai * HALF + m * 16) * MN + col0;
#pragma unroll
                        for (int bj = 0; bj < 2; ++bj)
#pragma unroll
                            for (int n = 0; n < 2; ++n) *(f32x4*)(rowp + bj * HALF + n * 16) = dead ? (f32x4){qn, qn, qn, qn} : (acc[ai][bj][m][n] * ox) * ow;
                    }
            }
        }
        STAMP(0x3000000 + (cur & 0xfffff));
        if (nxt < 0) break;
#pragma unroll
        for (int a = 0; a < 2; ++a)
#pragma unroll
            for (int b = 0; b < 2; ++b)
#pragma unroll
                for (int m = 0; m < 4; ++m)
#pragma unroll
                    for (int n = 0; n < 2; ++n) acc[a][b][m][n] = (f32x4){0.f, 0.f, 0.f, 0.f};
        cur = nxt; cA = Abase + (size_t)(cur & 255) * tstep; cB = Bbase + (size_t)((cur >> 8) & 255) * tstep;
    }
    G_WAIT_V(0);
    if (wr == 0) G_BAR;
    G_BAR;
}

__device__ __forceinline__ int scale_exp(unsigned bits) {
    const int e = (int)(bits >> 23);
    if (bits == 0u || e == 255) return 0;
    const int s = 13 - (e - 127);
    return s > 126 ? 126 : (s < -126 ? -126 : s);
}
__device__ __forceinline__ float pow2i(int s) { return __builtin_bit_cast(float, (unsigned)(s + 127) << 23); }
__device__ __forceinline__ void scale_phase(const Params& p, LAS unsigned char* lds, unsigned seq, int& esx, int& esw) {
    const int tid = threadIdx.x, lane = tid & 63, wid = tid >> 6;
    char* myws = p.ws[p.rank];
    volatile LAS unsigned* red = (volatile LAS unsigned*)lds;
    constexpr int N4 = MN * KSH / 4 / GRID;
    const u32x4* xs = (const u32x4*)p.x + (size_t)blockIdx.x * N4; const u32x4* wsrc = (const u32x4*)p.w + (size_t)blockIdx.x * N4;
    unsigned mx = 0, mw = 0;
#pragma unroll 4
    for (int i = tid; i < N4; i += 512) {
        const u32x4 a = xs[i], b = wsrc[i];
        mx = max(max(mx, a.x & 0x7fffffffu), max(max(a.y & 0x7fffffffu, a.z & 0x7fffffffu), a.w & 0x7fffffffu));
        mw = max(max(mw, b.x & 0x7fffffffu), max(max(b.y & 0x7fffffffu, b.z & 0x7fffffffu), b.w & 0x7fffffffu));
    }
#pragma unroll
    for (int o = 32; o >= 1; o >>= 1) { mx = max(mx, (unsigned)__shfl_xor((int)mx, o)); mw = max(mw, (unsigned)__shfl_xor((int)mw, o)); }
    if (lane == 0) { red[wid] = mx; red[8 + wid] = mw; }
    __syncthreads();
    if (tid == 0) {
        for (int i = 1; i < 8; ++i) { mx = max(mx, (unsigned)red[i]); mw = max(mw, (unsigned)red[8 + i]); }
        const unsigned long long tag = (unsigned long long)seq << 32;
        __hip_atomic_fetch_max((unsigned long long*)(myws + OFF_AMAX), tag | mx, __ATOMIC_RELAXED, __HIP_MEMORY_SCOPE_AGENT);
        __hip_atomic_fetch_max((unsigned long long*)(myws + OFF_AMAX + 64), tag | mw, __ATOMIC_RELAXED, __HIP_MEMORY_SCOPE_AGENT);
        asm volatile("s_waitcnt vmcnt(0)" ::: "memory");
        if ((__hip_atomic_fetch_add((unsigned*)(myws + OFF_AMAX + 256), 1u, __ATOMIC_RELAXED, __HIP_MEMORY_SCOPE_AGENT) % GRID) == (unsigned)(GRID - 1)) {
            const unsigned long long vx = __hip_atomic_load((unsigned long long*)(myws + OFF_AMAX), __ATOMIC_RELAXED, __HIP_MEMORY_SCOPE_AGENT);
            const unsigned long long vw = __hip_atomic_load((unsigned long long*)(myws + OFF_AMAX + 64), __ATOMIC_RELAXED, __HIP_MEMORY_SCOPE_AGENT);
#pragma unroll
            for (int q = 0; q < NR; ++q) {
                __hip_atomic_store((unsigned long long*)(p.ws[q] + OFF_SCALE + (size_t)p.rank * 128), vx, __ATOMIC_RELAXED, __HIP_MEMORY_SCOPE_SYSTEM);
                __hip_atomic_store((unsigned long long*)(p.ws[q] + OFF_SCALE + (size_t)p.rank * 128 + 8), vw, __ATOMIC_RELAXED, __HIP_MEMORY_SCOPE_SYSTEM);
            }
        }
    }
    if (wid == 0) {
        const unsigned long long* f = (const unsigned long long*)(myws + OFF_SCALE + (size_t)((lane >> 1) & 7) * 128 + (lane & 1) * 8);
        unsigned* tmo = (unsigned*)(myws + OFF_TMO);
        const unsigned long long t0 = __builtin_amdgcn_s_memrealtime();
        unsigned long long v;
        for (;;) {
            v = __hip_atomic_load(f, __ATOMIC_RELAXED, __HIP_MEMORY_SCOPE_SYSTEM);
            if (__all((unsigned)(v >> 32) == seq)) break;
            const unsigned tv = __hip_atomic_load(tmo, __ATOMIC_RELAXED, __HIP_MEMORY_SCOPE_AGENT);
            if (__builtin_amdgcn_readfirstlane(tv) == seq || __builtin_amdgcn_s_memrealtime() - t0 > DEADLINE) {
                if (lane == 0) __hip_atomic_store(tmo, seq, __ATOMIC_RELAXED, __HIP_MEMORY_SCOPE_AGENT);
                break;
            }
            __builtin_amdgcn_s_sleep(4);
        }
        unsigned m = (unsigned)v;
#pragma unroll
        for (int o = 2; o <= 8; o <<= 1) m = max(m, (unsigned)__shfl_xor((int)m, o));
        if (lane < 2) red[16 + lane] = (unsigned)scale_exp(m);
    }
    __syncthreads();
    esx = (int)red[16]; esw = (int)red[17];
    __syncthreads();
}

__global__ __launch_bounds__(512, 2) void fused_kernel(Params p) {
    extern __shared__ __attribute__((aligned(16))) unsigned char shm[];
    LAS unsigned char* lds = (LAS unsigned char*)shm;
    volatile LAS unsigned* sq = (volatile LAS unsigned*)(lds + STAGE_BYTES);
    if (threadIdx.x == 0) {
        const unsigned t = __hip_atomic_fetch_add((unsigned*)(p.ws[p.rank] + OFF_CALL), 1u, __ATOMIC_RELAXED, __HIP_MEMORY_SCOPE_AGENT);
        sq[0] = t / GRID + 1u;
    }
    __syncthreads();
    const unsigned seq = __builtin_amdgcn_readfirstlane(sq[0]);
    int nst = 0;
    STAMP(1);
    int esx, esw;
    scale_phase(p, lds, seq, esx, esw);
    esx = __builtin_amdgcn_readfirstlane(esx); esw = __builtin_amdgcn_readfirstlane(esw);
    STAMP(2);
    if ((int)blockIdx.x < NPUSH + NFWD) {
        if ((int)blockIdx.x < NPUSH) push_phase(p, (int)blockIdx.x, seq, pow2i(esx), pow2i(esw), nst);
        forward_phase(p, lds, (int)blockIdx.x, seq, nst);
    } else gemm_phase(p, lds, seq, pow2i(-esx), pow2i(-esw), nst);
}
}

extern "C" void kernel_launch(int rank, int nranks, ncclComm_t comm,
                              hipStream_t stream,
                              void* const* d_in, const int* in_sizes, int n_in,
                              void* d_out, int out_size,
                              const int* in_dtypes, int out_dtype,
                              void* const* d_out_all,
                              void* const* d_ws_all, size_t ws_bytes) {
    if (nranks != NR || ws_bytes < WS_NEED || n_in < 2) { fprintf(stderr, "unsupported configuration\n"); return; }
    static bool inits[NR] = {};
    static Params Ps[NR];
    if (rank < 0 || rank >= NR) return;
    bool& init = inits[rank];
    Params& P = Ps[rank];
    if (!init) {
        HIP_CHECK(hipFuncSetAttribute((const void*)fused_kernel, hipFuncAttributeMaxDynamicSharedMemorySize, LDS_BYTES));
        std::vector<int> own;
        for (int h = 0; h < 2; ++h) for (int pm = SHBASE; pm < NPAN; ++pm) if (sh_owner(pm, h * 16) == rank) for (int pn = h * 16; pn < h * 16 + 16; ++pn) own.push_back(pm | pn << 5);
        for (int i = 0; i < NOWN; ++i) P.own[i] = (unsigned short)own[i];
        for (int x = 0; x < NR; ++x) {
            std::vector<std::pair<int, int>> v;
            for (int i = 0; i < SHBASE / 2; ++i) for (int j = 0; j < 8; ++j) {
                const int pm = 2 * i + (x & 1), pn = 4 * j + (x >> 1);
                const int region = (pm < 16 ? 0 : 2) + (pn < 16 ? 0 : 1);
                v.push_back({region * 4096 + 1024 + j * 64 + i, pm | pn << 5});
            }
            for (int i = x; i < NOWN; i += NR) { const int pn = (own[i] >> 5) & 31; v.push_back({(pn < 16 ? 0 : 1) * 4096 + i, own[i] | (i + 1) << 10}); }
            std::sort(v.begin(), v.end());
            if ((int)v.size() != QTILES) { fprintf(stderr, "list size mismatch\n"); return; }
            for (int k = 0; k < QTILES; ++k) {
                const bool split = (k >= A0 && k < A0 + S0) || k >= QTILES - NSPLIT;
                if (split && (v[k].second >> 10)) { fprintf(stderr, "shared tile in a split segment\n"); return; }
                P.order[x][k] = (unsigned short)v[k].second;
            }
        }
        init = true;
    }
    P.x = (const float*)d_in[0]; P.w = (const float*)d_in[1]; P.out = (float*)d_out; P.rank = rank;
    for (int r = 0; r < NR; ++r) { P.ws[r] = (char*)d_ws_all[r]; P.outs[r] = (float*)d_out_all[r]; }
    hipLaunchKernelGGL(fused_kernel, dim3(GRID), dim3(512), LDS_BYTES, stream, P);
    HIP_CHECK(hipGetLastError());
}
```

```cpp
#include <hip/hip_runtime.h>
#include <rccl/rccl.h>
#include <cstdio>
#include <cstdint>
#include <algorithm>
#include <vector>

#define HIP_CHECK(expr) do { hipError_t _e = (expr); if (_e != hipSuccess) { \
    fprintf(stderr, "HIP error %s at %s:%d\n", hipGetErrorString(_e), __FILE__, __LINE__); \
    return; } } while (0)

namespace {
#define LAS __attribute__((address_space(3)))
typedef _Float16 f16x8 __attribute__((ext_vector_type(8)));
typedef float f32x4 __attribute__((ext_vector_type(4)));
typedef float f32x2 __attribute__((ext_vector_type(2)));
typedef unsigned u32x4 __attribute__((ext_vector_type(4)));

constexpr int NR = 8;
constexpr int MN = 8192;
constexpr int KSH = 1024;
constexpr int BM = 256, BK = 64, HALF = 128;
constexpr int HTB = HALF * BK * 2;
constexpr int NKT = NR * KSH / BK;
constexpr int KT_PER_RANK = KSH / BK;
constexpr int NPAN = MN / BM;
constexpr int NG = 4;
constexpr size_t MAT_BYTES = (size_t)MN * NR * KSH * 2;
constexpr size_t CTRL_BYTES = (size_t)1 << 20;
#define NSPLIT 4
#define NSH 12
static_assert(NSH % 4 == 0 && NSH <= 12, "NSH");
constexpr int SHBASE = NPAN - NSH;
constexpr int NOWN = NSH * NPAN / NR;
__host__ __device__ constexpr int sh_owner(int pm, int pn) { const int u = (pm - SHBASE) * 2 + (pn >> 4); return (u + (u >> 3)) & (NR - 1); }
constexpr int QTILES = (SHBASE * NPAN + NOWN) / NR;
#define A0 22
#define S0 12
constexpr int NMID = QTILES - A0 - S0 - NSPLIT;
constexpr int QLEN = A0 + S0 * NG + NMID + NSPLIT * NG;
constexpr int NSLOT = S0 + NSPLIT;
constexpr size_t TILE_BYTES = (size_t)BM * BM * 4;
constexpr size_t OFF_PART = CTRL_BYTES + 4 * MAT_BYTES;
constexpr size_t WS_NEED = OFF_PART + (size_t)NR * (S0 + NSPLIT) * NG * TILE_BYTES;
constexpr int STAGE_BYTES = 8 * HTB;
constexpr int LDS_BYTES = STAGE_BYTES + 64;
constexpr int GRID = 256;
#define NPUSH 32
#define NFWD 0
constexpr unsigned long long DEADLINE = 50000000ull;

constexpr size_t OFF_CALL = 0;
constexpr size_t OFF_TMO = 128;
constexpr size_t OFF_QUEUE = 1024;
constexpr size_t OFF_CENSUS = 4096;
#define FQ 2
constexpr size_t OFF_FWDQ = 3072;
constexpr size_t OFF_AMAX = 5120;
constexpr size_t OFF_SCALE = 6144;
constexpr size_t OFF_FLAG = 8192;
constexpr size_t OFF_GRP = 64 * 1024;
constexpr size_t OFF_CNT = 256 * 1024;
constexpr size_t OFF_DONE = 384 * 1024;
constexpr size_t OFF_OUTF = 448 * 1024;
__host__ __device__ constexpr size_t flag_off(int src, int reg, int g) { return OFF_FLAG + (size_t)((src * 3 + reg) * NG + g) * 128; }
constexpr unsigned FLAG_PER_CALL = NPUSH;

#define STAMP(tag) do { } while (0)
#define REL_SYS() __builtin_amdgcn_fence(__ATOMIC_RELEASE, "")
#define ST_REMOTE(ptr, val) asm volatile("global_store_dwordx4 %0, %1, off sc0 sc1" :: "v"(ptr), "v"(val) : "memory")
struct Params {
    const float* x; const float* w; float* out;
    char* ws[NR];
    float* outs[NR];
    int rank;
    unsigned short order[NR][128];
    unsigned short own[64];
};

__device__ __forceinline__ int lds_byte(int r, int c) { const int st = (r >> 4) * 2 + (c >> 5), rr = r & 15, cc = c & 31, ob = rr * 64 + cc * 2; return st * 1024 + (ob ^ (((ob >> 9) & 1) << 5)); }
__device__ __forceinline__ void stage_rc(int b, int& R, int& C) { const int st = b / 1024, sb = b % 1024, swz = sb ^ (((sb >> 9) & 1) << 5); R = (st >> 1) * 16 + swz / 64; C = (st & 1) * 32 + (swz % 64) / 2; }

__device__ __forceinline__ unsigned pack2(float a, float b, float sc) {
    const _Float16 ha = (_Float16)(a * sc), hb = (_Float16)(b * sc);
    return (unsigned)__builtin_bit_cast(unsigned short, ha) | ((unsigned)__builtin_bit_cast(unsigned short, hb) << 16);
}

__device__ __forceinline__ unsigned seq_inc(unsigned long long* q, unsigned seq, unsigned add) {
    unsigned long long old = __hip_atomic_load(q, __ATOMIC_RELAXED, __HIP_MEMORY_SCOPE_AGENT);
    for (;;) {
        const unsigned cnt = ((unsigned)(old >> 32) == seq) ? (unsigned)old : 0u;
        if (__hip_atomic_compare_exchange_strong(q, &old, ((unsigned long long)seq << 32) | (cnt + add), __ATOMIC_RELAXED, __ATOMIC_RELAXED, __HIP_MEMORY_SCOPE_AGENT)) return cnt;
    }
}
__device__ __forceinline__ unsigned seq_get(unsigned long long* q, unsigned seq) {
    const unsigned long long v = __hip_atomic_load(q, __ATOMIC_RELAXED, __HIP_MEMORY_SCOPE_AGENT);
    return ((unsigned)(v >> 32) == seq) ? (unsigned)v : 0u;
}

__device__ __forceinline__ void push_phase(const Params& p, int pw, unsigned seq, float sx, float sw, int& nst) {
    const int tid = threadIdx.x;
    const size_t par = (size_t)(seq & 1) * 2 * MAT_BYTES;
    int R[2], C[2];
    stage_rc(tid * 16, R[0], C[0]); stage_rc(tid * 16 + 8192, R[1], C[1]);
    const int wkg = tid & 7, wng = tid >> 3;
    char* dst[NR];
#pragma unroll
    for (int q = 0; q < NR; ++q) dst[q] = p.ws[(p.rank + q) & (NR - 1)];
    char* myws = p.ws[p.rank];
    const unsigned xcc = (unsigned)__builtin_amdgcn_s_getreg((3 << 11) | 20) & 7u;
    unsigned ngrp = 0;
    if (tid == 0) {
        seq_inc((unsigned long long*)(myws + OFF_CENSUS + xcc * 64), seq, 1u);
        seq_inc((unsigned long long*)(myws + OFF_CENSUS + 512), seq, 1u);
        const unsigned long long t0 = __builtin_amdgcn_s_memrealtime();
        while (seq_get((unsigned long long*)(myws + OFF_CENSUS + 512), seq) < (unsigned)NPUSH) {
            if (__builtin_amdgcn_s_memrealtime() - t0 > DEADLINE) { __hip_atomic_store((unsigned*)(myws + OFF_TMO), seq, __ATOMIC_RELAXED, __HIP_MEMORY_SCOPE_AGENT); break; }
            __builtin_amdgcn_s_sleep(1);
        }
        ngrp = seq_get((unsigned long long*)(myws + OFF_CENSUS + xcc * 64), seq);
    }
    for (int step = 0; step < 2 * NG + 1; ++step) {
        const int reg = step >> 2, g = step & 3;
        const int nitem = reg == 0 ? (2 * 16 + NSH) * 8 : reg == 1 ? 16 * 8 : (SHBASE - 16) * 32;
        for (int it0 = pw; it0 < nitem; it0 += 2 * NPUSH) {
            u32x4 o[2][2]; size_t off[2][2]; unsigned mask[2];
            f32x4 a[2][2][2]; f32x2 bq[2][8]; int mats[2];
#pragma unroll
            for (int z = 0; z < 2; ++z) {
                const int it = it0 + z * NPUSH;
                mask[z] = it < nitem ? 0xffu : 0u;
                const int itc = it < nitem ? it : it0;
                const int hb = itc & 1, i2 = itc >> 1;
                const int pi = reg == 2 ? i2 >> 4 : i2 >> 2, kt = reg == 2 ? (i2 & 15) : g * 4 + (i2 & 3);
                int mat, j;
                if (reg == 0) { if (pi < 16) { mat = 0; j = pi; } else if (pi < 32) { mat = 1; j = pi - 16; }
                                else { mat = 0; j = SHBASE + pi - 32; mask[z] &= (1u << sh_owner(j, 0)) | (1u << sh_owner(j, 16)); } }
                else if (reg == 1) { mat = 1; j = 16 + pi; }
                else { mat = 0; j = 16 + pi; }
                mats[z] = mat;
                const size_t ioff = CTRL_BYTES + par + (size_t)mat * MAT_BYTES + ((size_t)(j * 2 + hb) * NKT + p.rank * KT_PER_RANK + kt) * HTB;
                if (mat == 0) {
#pragma unroll
                    for (int v = 0; v < 2; ++v) {
                        const float* sp = p.x + (size_t)(j * BM + hb * HALF + R[v]) * KSH + kt * BK + C[v];
                        a[z][v][0] = *(const f32x4*)sp; a[z][v][1] = *(const f32x4*)(sp + 4);
                        off[z][v] = ioff + v * 8192 + tid * 16;
                    }
                } else {
                    const float* sp = p.w + (size_t)(kt * BK + wkg * 8) * MN + j * BM + hb * HALF + wng * 2;
#pragma unroll
                    for (int e = 0; e < 8; ++e) bq[z][e] = *(const f32x2*)(sp + (size_t)e * MN);
                    off[z][0] = ioff + lds_byte(wng * 2, wkg * 8); off[z][1] = ioff + lds_byte(wng * 2 + 1, wkg * 8);
                }
            }
#pragma unroll
            for (int z = 0; z < 2; ++z) {
                if (mats[z] == 0) {
#pragma unroll
                    for (int v = 0; v < 2; ++v) {
                        o[z][v].x = pack2(a[z][v][0][0], a[z][v][0][1], sx); o[z][v].y = pack2(a[z][v][0][2], a[z][v][0][3], sx);
                        o[z][v].z = pack2(a[z][v][1][0], a[z][v][1][1], sx); o[z][v].w = pack2(a[z][v][1][2], a[z][v][1][3], sx);
                    }
                } else {
#pragma unroll
                    for (int e = 0; e < 2; ++e) {
                        o[z][e].x = pack2(bq[z][0][e], bq[z][1][e], sw); o[z][e].y = pack2(bq[z][2][e], bq[z][3][e], sw);
                        o[z][e].z = pack2(bq[z][4][e], bq[z][5][e], sw); o[z][e].w = pack2(bq[z][6][e], bq[z][7][e], sw);
                    }
                }
#pragma unroll
                for (int q = 0; q < NR; ++q) {
                    if (!((mask[z] >> ((p.rank + q) & (NR - 1))) & 1u)) continue;
#pragma unroll
                    for (int v = 0; v < 2; ++v) ST_REMOTE(dst[q] + off[z][v], o[z][v]);
                }
            }
        }
        asm volatile("s_waitcnt vmcnt(0)" ::: "memory");
        __builtin_amdgcn_s_barrier();
        if (tid == 0) {
            const unsigned old = seq_inc((unsigned long long*)(myws + OFF_GRP + (size_t)(xcc * 16 + step) * 64), seq, 1u);
            if (old + 1u == ngrp) {
                REL_SYS();
                asm volatile("s_waitcnt vmcnt(0)" ::: "memory");
                for (int gg = (reg == 2 ? 0 : g); gg <= (reg == 2 ? NG - 1 : g); ++gg)
#pragma unroll
                    for (int q = 0; q < NR; ++q)
                        __hip_atomic_fetch_add((unsigned*)(dst[q] + flag_off(p.rank, reg, gg)), ngrp, __ATOMIC_RELAXED, __HIP_MEMORY_SCOPE_SYSTEM);
            }
        }
        STAMP(0x1000000 + step);
    }
}

__device__ __forceinline__ void forward_phase(const Params& p, LAS unsigned char* lds, int pw, unsigned seq, int& nst) {
    const int tid = threadIdx.x;
    char* myws = p.ws[p.rank];
    unsigned* tmo = (unsigned*)(myws + OFF_TMO);
    volatile LAS int* fb = (volatile LAS int*)(lds + 64);
    for (;;) {
        __syncthreads();
        if (tid == 0) fb[0] = (int)seq_inc((unsigned long long*)(myws + OFF_FWDQ), seq, 1u);
        __syncthreads();
        const int iu = fb[0];
        if (iu >= NOWN * FQ) break;
        const int i = iu / FQ, qr = iu % FQ;
        if (tid == 0) {
            const unsigned* d = (const unsigned*)(myws + OFF_DONE + (size_t)i * 64);
            const unsigned long long t0 = __builtin_amdgcn_s_memrealtime();
            while ((int)(__hip_atomic_load(d, __ATOMIC_RELAXED, __HIP_MEMORY_SCOPE_AGENT) - seq * 8u) < 0) {
                if (__hip_atomic_load(tmo, __ATOMIC_RELAXED, __HIP_MEMORY_SCOPE_AGENT) == seq || __builtin_amdgcn_s_memrealtime() - t0 > DEADLINE) {
                    __hip_atomic_store(tmo, seq, __ATOMIC_RELAXED, __HIP_MEMORY_SCOPE_AGENT); break; }
                __builtin_amdgcn_s_sleep(16);
            }
            __builtin_amdgcn_fence(__ATOMIC_ACQUIRE, "agent");
            asm volatile("s_waitcnt vmcnt(0)" ::: "memory");
        }
        __syncthreads();
        const int t = p.own[i];
        const size_t base = ((size_t)(t & 31) * BM) * MN + (size_t)((t >> 5) & 31) * BM;
        for (int r0 = qr * (BM / FQ); r0 < (qr + 1) * (BM / FQ); r0 += 32) {
            f32x4 v[4]; size_t off[4];
#pragma unroll
            for (int u = 0; u < 4; ++u) { off[u] = base + (size_t)(r0 + u * 8 + (tid >> 6)) * MN + (tid & 63) * 4; v[u] = *(const f32x4*)(p.out + off[u]); }
#pragma unroll
            for (int q = 1; q < NR; ++q) {
                float* o = p.outs[(p.rank + q) & (NR - 1)];
#pragma unroll
                for (int u = 0; u < 4; ++u) ST_REMOTE(o + off[u], v[u]);
            }
        }
        asm volatile("s_waitcnt vmcnt(0)" ::: "memory");
        __syncthreads();
        if (tid == 0) {
            REL_SYS();
            asm volatile("s_waitcnt vmcnt(0)" ::: "memory");
#pragma unroll
            for (int q = 1; q < NR; ++q)
                __hip_atomic_fetch_add((unsigned*)(p.ws[(p.rank + q) & (NR - 1)] + OFF_OUTF + (size_t)p.rank * 128), 1u, __ATOMIC_RELAXED, __HIP_MEMORY_SCOPE_SYSTEM);
        }
        STAMP(0x4000000 + iu);
    }
    if (pw == 0 && tid < 64) {
        const int src = tid & 7;
        const unsigned* f = (const unsigned*)(myws + OFF_OUTF + (size_t)src * 128);
        const unsigned want = seq * (unsigned)(NOWN * FQ);
        const unsigned long long t0 = __builtin_amdgcn_s_memrealtime();
        bool dead = false;
        for (;;) {
            const unsigned v = __hip_atomic_load(f, __ATOMIC_RELAXED, __HIP_MEMORY_SCOPE_SYSTEM);
            if (__all(src == p.rank || (int)(v - want) >= 0)) break;
            const unsigned tv = __hip_atomic_load(tmo, __ATOMIC_RELAXED, __HIP_MEMORY_SCOPE_AGENT);
            if (__builtin_amdgcn_readfirstlane(tv) == seq || __builtin_amdgcn_s_memrealtime() - t0 > DEADLINE) { dead = true; break; }
            __builtin_amdgcn_s_sleep(16);
        }
        __builtin_amdgcn_fence(__ATOMIC_ACQUIRE, "");
        if (dead && tid == 0) { __hip_atomic_store(tmo, seq, __ATOMIC_RELAXED, __HIP_MEMORY_SCOPE_AGENT); p.out[0] = __builtin_nanf(""); }
        STAMP(0x5000000);
    }
}

__device__ __forceinline__ void wait_group(const Params& p, unsigned seq, int code, int g, int lane, bool& dead) {
    char* myws = p.ws[p.rank];
    const int pm = code & 255, pn = (code >> 8) & 255;
    const int reg = (lane & 1) ? (pn < NPAN / 2 ? 0 : 1) : ((pm < NPAN / 2 || pm >= SHBASE) ? 0 : 2);
    const unsigned* f = (const unsigned*)(myws + flag_off((lane >> 1) & 7, reg, g));
    const unsigned want = seq * FLAG_PER_CALL;
    unsigned* tmo = (unsigned*)(myws + OFF_TMO);
    const unsigned long long t0 = __builtin_amdgcn_s_memrealtime();
    for (;;) {
        const unsigned v = __hip_atomic_load(f, __ATOMIC_RELAXED, __HIP_MEMORY_SCOPE_SYSTEM);
        if (__all((int)(v - want) >= 0)) break;
        const unsigned tv = __hip_atomic_load(tmo, __ATOMIC_RELAXED, __HIP_MEMORY_SCOPE_AGENT);
        if (__builtin_amdgcn_readfirstlane(tv) == seq || __builtin_amdgcn_s_memrealtime() - t0 > DEADLINE) {
            if (lane == 0) __hip_atomic_store(tmo, seq, __ATOMIC_RELAXED, __HIP_MEMORY_SCOPE_AGENT);
            dead = true; break;
        }
        __builtin_amdgcn_s_sleep(16);
    }
    __builtin_amdgcn_fence(__ATOMIC_ACQUIRE, "wavefront");
    asm volatile("s_waitcnt vmcnt(0)" ::: "memory");
}

__device__ __forceinline__ int draw_unit(const Params& p, unsigned seq, unsigned xcc, int lane) {
    char* myws = p.ws[p.rank];
    int code = -1;
    if (lane == 0) {
        for (int s = 0; s < NR && code < 0; ++s) {
            const unsigned q = (xcc + s) & (NR - 1);
            unsigned long long* qp = (unsigned long long*)(myws + OFF_QUEUE + q * 128);
            unsigned long long old = __hip_atomic_load(qp, __ATOMIC_RELAXED, __HIP_MEMORY_SCOPE_AGENT);
            for (;;) {
                const unsigned cnt = ((unsigned)(old >> 32) == seq) ? (unsigned)old : 0u;
                if (cnt >= (unsigned)QLEN) break;
                const unsigned long long nw = ((unsigned long long)seq << 32) | (cnt + 1u);
                if (__hip_atomic_compare_exchange_strong(qp, &old, nw, __ATOMIC_RELAXED, __ATOMIC_RELAXED, __HIP_MEMORY_SCOPE_AGENT)) {
                    int c = (int)cnt, ti, kq = -1, slot = 0;
                    if (c < A0) ti = c;
                    else if ((c -= A0) < S0 * NG) { constexpr int S0D = S0 > 0 ? S0 : 1; ti = A0 + c % S0D; kq = c / S0D; slot = c % S0D; }
                    else if ((c -= S0 * NG) < NMID) ti = A0 + S0 + c;
                    else { c -= NMID; ti = A0 + S0 + NMID + c % NSPLIT; kq = c / NSPLIT; slot = S0 + c % NSPLIT; }
                    const int t = p.order[q][ti];
                    code = (t & 31) | (((t >> 5) & 31) << 8);
                    if (kq >= 0) code |= (kq << 16) | (1 << 18) | (((int)q * NSLOT + slot) << 20);
                    else if (t >> 10) code |= (1 << 19) | (((t >> 10) - 1) << 20);
                    break;
                }
            }
        }
    }
    return __builtin_amdgcn_readfirstlane(code);
}

__device__ __forceinline__ void gemm_phase(const Params& p, LAS unsigned char* lds, unsigned seq, float ox, float ow, int& nst) {
    const int tid = threadIdx.x, wid = __builtin_amdgcn_readfirstlane(tid >> 6), lane = tid & 63, wr = wid >> 2, wc = wid & 3, fr = lane & 15, fq = lane >> 4;
    const unsigned xcc = (unsigned)__builtin_amdgcn_s_getreg((3 << 11) | 20) & 7u;
    volatile LAS int* bc = (volatile LAS int*)(lds + STAGE_BYTES + 16);
    const char* Abase = p.ws[p.rank] + CTRL_BYTES + (size_t)(seq & 1) * 2 * MAT_BYTES;
    const char* Bbase = Abase + MAT_BYTES;
    constexpr size_t hstep = (size_t)NKT * HTB;
    constexpr size_t tstep = 2 * hstep;
    const unsigned voff = (unsigned)tid * 16u;
    const unsigned ldsw = (unsigned)wid * 1024u;
    const int aoff = lds_byte(wr * 64 + fr, fq * 8), boff = lds_byte(wc * 32 + fr, fq * 8);
    const int rank = p.rank;
    bool dead = false;
#define KOFF(T) ((size_t)((((rank + (((T) >> 2) & 7)) & 7) << 4) | (((T) >> 5) << 2) | ((T) & 3)) * HTB)
#define U_T0(c) ((((c) >> 18) & 1) * (((c) >> 16) & 3) * 32)
#define U_NT(c) ((((c) >> 18) & 1) ? 32 : NKT)
#define G_SA(b, h) (((b) * 2 + (h)) * HTB)
#define G_SB(b, h) ((4 + (b) * 2 + (h)) * HTB)
#define G_STAGE(bufoff, gbase) do { _Pragma("unroll") for (int _i = 0; _i < 2; ++_i) \
        __builtin_amdgcn_global_load_lds((const unsigned*)((const char*)(gbase) + voff + _i * 8192), (LAS unsigned*)(lds + (bufoff) + ldsw + _i * 8192), 16, 0, 0); } while (0)
#define G_LDA(dst, b, h) do { _Pragma("unroll") for (int m = 0; m < 4; ++m) _Pragma("unroll") for (int k = 0; k < 2; ++k) dst[m][k] = *(const LAS f16x8*)(lds + G_SA(b, h) + aoff + m * 2048 + k * 1024); } while (0)
#define G_LDB(dst, b, h) do { _Pragma("unroll") for (int n = 0; n < 2; ++n) _Pragma("unroll") for (int k = 0; k < 2; ++k) dst[n][k] = *(const LAS f16x8*)(lds + G_SB(b, h) + boff + n * 2048 + k * 1024); } while (0)
#define G_MMA(ai, bj, At, Bt) do { __builtin_amdgcn_s_setprio(1); _Pragma("unroll") for (int m = 0; m < 4; ++m) _Pragma("unroll") for (int n = 0; n < 2; ++n) _Pragma("unroll") for (int k = 0; k < 2; ++k) \
        acc[ai][bj][m][n] = __builtin_amdgcn_mfma_f32_16x16x32_f16(Bt[n][k], At[m][k], acc[ai][bj][m][n], 0, 0, 0); __builtin_amdgcn_s_setprio(0); } while (0)
#define G_WAIT_V(n) asm volatile("s_waitcnt vmcnt(" #n ")" ::: "memory")
#define G_WAIT_L(n) asm volatile("s_waitcnt lgkmcnt(" #n ")" ::: "memory")
#define G_BAR __builtin_amdgcn_s_barrier()
#define G_SCHED __builtin_amdgcn_sched_barrier(0)
#define G_NEXT(codevar) do { if (wid == 0) { bool _d = false; const int _c = draw_unit(p, seq, xcc, lane); if (_c >= 0) wait_group(p, seq, _c, U_T0(_c) >> 5, lane, _d); \
            if (lane == 0) { bc[0] = _c; if (_d) bc[1] = 1; } STAMP(0x2000000 + (_c & 0xfffff)); asm volatile("s_waitcnt lgkmcnt(0)" ::: "memory"); } \
        asm volatile("" ::: "memory"); G_BAR; asm volatile("" ::: "memory"); codevar = __builtin_amdgcn_readfirstlane(bc[0]); } while (0)
#define G_GROUP(code, g) do { if (wid == 0) { bool _d = false; STAMP(0x6000000 + (g)); wait_group(p, seq, code, g, lane, _d); STAMP(0x7000000 + (g)); if (lane == 0 && _d) bc[1] = 1; asm volatile("s_waitcnt lgkmcnt(0)" ::: "memory"); } \
        asm volatile("" ::: "memory"); G_BAR; asm volatile("" ::: "memory"); } while (0)

    if (tid == 0) { bc[0] = -1; bc[1] = 0; __builtin_amdgcn_fence(__ATOMIC_ACQUIRE, ""); asm volatile("s_waitcnt vmcnt(0)" ::: "memory"); }
    __syncthreads();
    int cur;
    G_NEXT(cur);
    if (cur < 0) return;
    f32x4 acc[2][2][4][2];
#pragma unroll
    for (int a = 0; a < 2; ++a)
#pragma unroll
        for (int b = 0; b < 2; ++b)
#pragma unroll
            for (int m = 0; m < 4; ++m)
#pragma unroll
                for (int n = 0; n < 2; ++n) acc[a][b][m][n] = (f32x4){0.f, 0.f, 0.f, 0.f};
    f16x8 At[4][2], B0[2][2], B1[2][2];
    const char* cA = Abase + (size_t)(cur & 255) * tstep; const char* cB = Bbase + (size_t)((cur >> 8) & 255) * tstep;
    {
        const size_t k0 = KOFF(U_T0(cur)), k1 = KOFF(U_T0(cur) + 1);
        G_STAGE(G_SB(0, 0), cB + k0); G_STAGE(G_SA(0, 0), cA + k0); G_STAGE(G_SB(0, 1), cB + hstep + k0); G_STAGE(G_SA(0, 1), cA + hstep + k0);
        if (wr == 1) G_BAR;
        G_WAIT_V(4); G_BAR;
        G_STAGE(G_SB(1, 0), cB + k1); G_STAGE(G_SA(1, 0), cA + k1); G_STAGE(G_SB(1, 1), cB + hstep + k1);
        G_WAIT_V(6); G_BAR;
    }
    for (;;) {
        int nxt = -1;
        const int tb = U_T0(cur), te = tb + U_NT(cur);
        for (int t = tb; t < te; t += 2) {
            const bool last = (t == te - 2);
            const char* a2; const char* b2; const char* a3; const char* b3;
            if (last) {
                G_NEXT(nxt);
                const int u = nxt >= 0 ? nxt : cur;
                const char* nA = Abase + (size_t)(u & 255) * tstep; const char* nB = Bbase + (size_t)((u >> 8) & 255) * tstep;
                const size_t k0 = KOFF(U_T0(u)), k1 = KOFF(U_T0(u) + 1);
                a2 = nA + k0; b2 = nB + k0; a3 = nA + k1; b3 = nB + k1;
            } else {
                if (((t + 2) & 31) == 0) G_GROUP(cur, (t + 2) >> 5);
                const size_t k2 = KOFF(t + 2), k3 = KOFF(t + 3);
                a2 = cA + k2; b2 = cB + k2; a3 = cA + k3; b3 = cB + k3;
            }
            const char* a1 = cA + KOFF(t + 1);
            G_LDB(B0, 0, 0); G_SCHED; G_LDA(At, 0, 0); G_STAGE(G_SA(1, 1), a1 + hstep);
            G_WAIT_L(8); G_BAR; G_WAIT_L(0); G_MMA(0, 0, At, B0); G_BAR; G_SCHED;
            G_LDB(B1, 0, 1); G_STAGE(G_SB(0, 0), b2);
            G_BAR; G_WAIT_L(0); G_MMA(0, 1, At, B1); G_BAR;
            G_LDA(At, 0, 1); G_STAGE(G_SA(0, 0), a2);
            G_BAR; G_WAIT_L(0); G_MMA(1, 0, At, B0); G_BAR; G_SCHED;
            G_STAGE(G_SB(0, 1), b2 + hstep);
            G_WAIT_V(6); G_BAR; G_MMA(1, 1, At, B1); G_BAR;
            G_LDB(B0, 1, 0); G_SCHED; G_LDA(At, 1, 0); G_STAGE(G_SA(0, 1), a2 + hstep);
            G_WAIT_L(8); G_BAR; G_WAIT_L(0); G_MMA(0, 0, At, B0); G_BAR; G_SCHED;
            G_LDB(B1, 1, 1); G_STAGE(G_SB(1, 0), b3);
            G_BAR; G_WAIT_L(0); G_MMA(0, 1, At, B1); G_BAR;
            G_LDA(At, 1, 1); G_STAGE(G_SA(1, 0), a3);
            G_BAR; G_WAIT_L(0); G_MMA(1, 0, At, B0); G_BAR; G_SCHED;
            G_STAGE(G_SB(1, 1), b3 + hstep);
            G_WAIT_V(6); G_BAR; G_MMA(1, 1, At, B1); G_BAR;
        }
        {
            dead = (bc[1] != 0);
            bool fin = true;
            if (cur & (1 << 18)) {
                const int slot = cur >> 20, kq = (cur >> 16) & 3;
                char* pbase = p.ws[p.rank] + OFF_PART + (size_t)slot * NG * TILE_BYTES + (size_t)tid * 16;
                f32x4* mine = (f32x4*)(pbase + (size_t)kq * TILE_BYTES);
#pragma unroll
                for (int i = 0; i < 32; ++i)
                    asm volatile("global_store_dwordx4 %0, %1, off sc1\n\ts_nop 1" :: "v"(mine + i * 512), "v"(acc[i >> 4][(i >> 3) & 1][(i >> 1) & 3][i & 1]) : "memory");
                asm volatile("s_waitcnt vmcnt(0)" ::: "memory");
                unsigned tk = 0;
                if (lane == 0) tk = __hip_atomic_fetch_add((unsigned*)(p.ws[p.rank] + OFF_CNT + (size_t)(slot * 8 + wid) * 64), 1u, __ATOMIC_RELAXED, __HIP_MEMORY_SCOPE_AGENT);
                tk = __builtin_amdgcn_readfirstlane(tk);
                fin = ((tk & (NG - 1)) == NG - 1);
                if (fin) {
                    __builtin_amdgcn_fence(__ATOMIC_ACQUIRE, "agent");
                    asm volatile("s_waitcnt vmcnt(0)" ::: "memory");
                    const f32x4* o1 = (const f32x4*)(pbase + (size_t)((kq + 1) & (NG - 1)) * TILE_BYTES);
                    const f32x4* o2 = (const f32x4*)(pbase + (size_t)((kq + 2) & (NG - 1)) * TILE_BYTES);
                    const f32x4* o3 = (const f32x4*)(pbase + (size_t)((kq + 3) & (NG - 1)) * TILE_BYTES);
#pragma unroll
                    for (int i0 = 0; i0 < 32; i0 += 8) {
                        f32x4 t1[8], t2[8], t3[8];
#pragma unroll
                        for (int i = 0; i < 8; ++i) { t1[i] = o1[(i0 + i) * 512]; t2[i] = o2[(i0 + i) * 512]; t3[i] = o3[(i0 + i) * 512]; }
#pragma unroll
                        for (int i = 0; i < 8; ++i) { const int ii = i0 + i; acc[ii >> 4][(ii >> 3) & 1][(ii >> 1) & 3][ii & 1] += (t1[i] + t2[i]) + t3[i]; }
                    }
                    const unsigned tv = __hip_atomic_load((unsigned*)(p.ws[p.rank] + OFF_TMO), __ATOMIC_RELAXED, __HIP_MEMORY_SCOPE_AGENT);
                    if (__builtin_amdgcn_readfirstlane(tv) == seq) dead = true;
                }
            }
            if (cur & (1 << 19)) {
                const float qn = __builtin_nanf("");
                const int row0 = (cur & 255) * BM + wr * 64 + fr, col0 = ((cur >> 8) & 255) * BM + wc * 32 + 4 * fq;
#pragma unroll
                for (int ai = 0; ai < 2; ++ai)
#pragma unroll
                    for (int m = 0; m < 4; ++m) {
                        float* rowp = p.out + (size_t)(row0 + ai * HALF + m * 16) * MN + col0;
#pragma unroll
                        for (int bj = 0; bj < 2; ++bj)
#pragma unroll
                            for (int n = 0; n < 2; ++n) { const f32x4 v = dead ? (f32x4){qn, qn, qn, qn} : (acc[ai][bj][m][n] * ox) * ow;
                                asm volatile("global_store_dwordx4 %0, %1, off sc1\n\ts_nop 1" :: "v"(rowp + bj * HALF + n * 16), "v"(v) : "memory"); }
                    }
                asm volatile("s_waitcnt vmcnt(0)" ::: "memory");
                if (lane == 0) __hip_atomic_fetch_add((unsigned*)(p.ws[p.rank] + OFF_DONE + (size_t)(cur >> 20) * 64), 1u, __ATOMIC_RELAXED, __HIP_MEMORY_SCOPE_AGENT);
            } else if (fin) {
                const float qn = __builtin_nanf("");
                const int row0 = (cur & 255) * BM + wr * 64 + fr, col0 = ((cur >> 8) & 255) * BM + wc * 32 + 4 * fq;
#pragma unroll
                for (int ai = 0; ai < 2; ++ai)
#pragma unroll
                    for (int m = 0; m < 4; ++m) {
                        float* rowp = p.out + (size_t)(row0 + ai * HALF + m * 16) * MN + col0;
#pragma unroll
                        for (int bj = 0; bj < 2; ++bj)
#pragma unroll
                            for (int n = 0; n < 2; ++n) *(f32x4*)(rowp + bj * HALF + n * 16) = dead ? (f32x4){qn, qn, qn, qn} : (acc[ai][bj][m][n] * ox) * ow;
                    }
            }
        }
        STAMP(0x3000000 + (cur & 0xfffff));
        if (nxt < 0) break;
#pragma unroll
        for (int a = 0; a < 2; ++a)
#pragma unroll
            for (int b = 0; b < 2; ++b)
#pragma unroll
                for (int m = 0; m < 4; ++m)
#pragma unroll
                    for (int n = 0; n < 2; ++n) acc[a][b][m][n] = (f32x4){0.f, 0.f, 0.f, 0.f};
        cur = nxt; cA = Abase + (size_t)(cur & 255) * tstep; cB = Bbase + (size_t)((cur >> 8) & 255) * tstep;
    }
    G_WAIT_V(0);
    if (wr == 0) G_BAR;
    G_BAR;
}

__device__ __forceinline__ int scale_exp(unsigned bits) {
    const int e = (int)(bits >> 23);
    if (bits == 0u || e == 255) return 0;
    const int s = 13 - (e - 127);
    return s > 126 ? 126 : (s < -126 ? -126 : s);
}
__device__ __forceinline__ float pow2i(int s) { return __builtin_bit_cast(float, (unsigned)(s + 127) << 23); }
__device__ __forceinline__ void scale_phase(const Params& p, LAS unsigned char* lds, unsigned seq, int& esx, int& esw) {
    const int tid = threadIdx.x, lane = tid & 63, wid = tid >> 6;
    char* myws = p.ws[p.rank];
    volatile LAS unsigned* red = (volatile LAS unsigned*)lds;
    constexpr int N4 = MN * KSH / 4 / GRID;
    for (;;) {
        if (tid == 0) red[18] = __hip_atomic_fetch_add((unsigned*)(myws + OFF_AMAX + 320), 1u, __ATOMIC_RELAXED, __HIP_MEMORY_SCOPE_AGENT) % (2u * GRID);
        __syncthreads();
        const unsigned chunk = red[18];
        if (chunk >= (unsigned)GRID) break;
        const u32x4* xs = (const u32x4*)p.x + (size_t)chunk * N4; const u32x4* wsrc = (const u32x4*)p.w + (size_t)chunk * N4;
        unsigned mx = 0, mw = 0;
#pragma unroll 4
        for (int i = tid; i < N4; i += 512) {
            const u32x4 a = xs[i], b = wsrc[i];
            mx = max(max(mx, a.x & 0x7fffffffu), max(max(a.y & 0x7fffffffu, a.z & 0x7fffffffu), a.w & 0x7fffffffu));
            mw = max(max(mw, b.x & 0x7fffffffu), max(max(b.y & 0x7fffffffu, b.z & 0x7fffffffu), b.w & 0x7fffffffu));
        }
#pragma unroll
        for (int o = 32; o >= 1; o >>= 1) { mx = max(mx, (unsigned)__shfl_xor((int)mx, o)); mw = max(mw, (unsigned)__shfl_xor((int)mw, o)); }
        if (lane == 0) { red[wid] = mx; red[8 + wid] = mw; }
        __syncthreads();
        if (tid == 0) {
            for (int i = 1; i < 8; ++i) { mx = max(mx, (unsigned)red[i]); mw = max(mw, (unsigned)red[8 + i]); }
            const unsigned long long tag = (unsigned long long)seq << 32;
            __hip_atomic_fetch_max((unsigned long long*)(myws + OFF_AMAX), tag | mx, __ATOMIC_RELAXED, __HIP_MEMORY_SCOPE_AGENT);
            __hip_atomic_fetch_max((unsigned long long*)(myws + OFF_AMAX + 64), tag | mw, __ATOMIC_RELAXED, __HIP_MEMORY_SCOPE_AGENT);
            asm volatile("s_waitcnt vmcnt(0)" ::: "memory");
            if ((__hip_atomic_fetch_add((unsigned*)(myws + OFF_AMAX + 256), 1u, __ATOMIC_RELAXED, __HIP_MEMORY_SCOPE_AGENT) % GRID) == (unsigned)(GRID - 1)) {
                const unsigned long long vx = __hip_atomic_load((unsigned long long*)(myws + OFF_AMAX), __ATOMIC_RELAXED, __HIP_MEMORY_SCOPE_AGENT);
                const unsigned long long vw = __hip_atomic_load((unsigned long long*)(myws + OFF_AMAX + 64), __ATOMIC_RELAXED, __HIP_MEMORY_SCOPE_AGENT);
#pragma unroll
                for (int q = 0; q < NR; ++q) {
                    __hip_atomic_store((unsigned long long*)(p.ws[q] + OFF_SCALE + (size_t)p.rank * 128), vx, __ATOMIC_RELAXED, __HIP_MEMORY_SCOPE_SYSTEM);
                    __hip_atomic_store((unsigned long long*)(p.ws[q] + OFF_SCALE + (size_t)p.rank * 128 + 8), vw, __ATOMIC_RELAXED, __HIP_MEMORY_SCOPE_SYSTEM);
                }
            }
        }
        __syncthreads();
    }
    if (wid == 0) {
        const unsigned long long* f = (const unsigned long long*)(myws + OFF_SCALE + (size_t)((lane >> 1) & 7) * 128 + (lane & 1) * 8);
        unsigned* tmo = (unsigned*)(myws + OFF_TMO);
        const unsigned long long t0 = __builtin_amdgcn_s_memrealtime();
        unsigned long long v;
        for (;;) {
            v = __hip_atomic_load(f, __ATOMIC_RELAXED, __HIP_MEMORY_SCOPE_SYSTEM);
            if (__all((unsigned)(v >> 32) == seq)) break;
            const unsigned tv = __hip_atomic_load(tmo, __ATOMIC_RELAXED, __HIP_MEMORY_SCOPE_AGENT);
            if (__builtin_amdgcn_readfirstlane(tv) == seq || __builtin_amdgcn_s_memrealtime() - t0 > DEADLINE) {
                if (lane == 0) __hip_atomic_store(tmo, seq, __ATOMIC_RELAXED, __HIP_MEMORY_SCOPE_AGENT);
                break;
            }
            __builtin_amdgcn_s_sleep(4);
        }
        unsigned m = (unsigned)v;
#pragma unroll
        for (int o = 2; o <= 8; o <<= 1) m = max(m, (unsigned)__shfl_xor((int)m, o));
        if (lane < 2) red[16 + lane] = (unsigned)scale_exp(m);
    }
    __syncthreads();
    esx = (int)red[16]; esw = (int)red[17];
    __syncthreads();
}

__global__ __launch_bounds__(512, 2) void fused_kernel(Params p) {
    extern __shared__ __attribute__((aligned(16))) unsigned char shm[];
    LAS unsigned char* lds = (LAS unsigned char*)shm;
    volatile LAS unsigned* sq = (volatile LAS unsigned*)(lds + STAGE_BYTES);
    if (threadIdx.x == 0) {
        const unsigned t = __hip_atomic_fetch_add((unsigned*)(p.ws[p.rank] + OFF_CALL), 1u, __ATOMIC_RELAXED, __HIP_MEMORY_SCOPE_AGENT);
        sq[0] = t / GRID + 1u;
    }
    __syncthreads();
    const unsigned seq = __builtin_amdgcn_readfirstlane(sq[0]);
    int nst = 0;
    STAMP(1);
    int esx, esw;
    scale_phase(p, lds, seq, esx, esw);
    esx = __builtin_amdgcn_readfirstlane(esx); esw = __builtin_amdgcn_readfirstlane(esw);
    STAMP(2);
    if ((int)blockIdx.x < NPUSH + NFWD) {
        if ((int)blockIdx.x < NPUSH) push_phase(p, (int)blockIdx.x, seq, pow2i(esx), pow2i(esw), nst);
        forward_phase(p, lds, (int)blockIdx.x, seq, nst);
    } else gemm_phase(p, lds, seq, pow2i(-esx), pow2i(-esw), nst);
}
}

extern "C" void kernel_launch(int rank, int nranks, ncclComm_t comm,
                              hipStream_t stream,
                              void* const* d_in, const int* in_sizes, int n_in,
                              void* d_out, int out_size,
                              const int* in_dtypes, int out_dtype,
                              void* const* d_out_all,
                              void* const* d_ws_all, size_t ws_bytes) {
    if (nranks != NR || ws_bytes < WS_NEED || n_in < 2) { fprintf(stderr, "unsupported configuration\n"); return; }
    static bool inits[NR] = {};
    static Params Ps[NR];
    if (rank < 0 || rank >= NR) return;
    bool& init = inits[rank];
    Params& P = Ps[rank];
    if (!init) {
        HIP_CHECK(hipFuncSetAttribute((const void*)fused_kernel, hipFuncAttributeMaxDynamicSharedMemorySize, LDS_BYTES));
        std::vector<int> own;
        for (int h = 0; h < 2; ++h) for (int pm = SHBASE; pm < NPAN; ++pm) if (sh_owner(pm, h * 16) == rank) for (int pn = h * 16; pn < h * 16 + 16; ++pn) own.push_back(pm | pn << 5);
        for (int i = 0; i < NOWN; ++i) P.own[i] = (unsigned short)own[i];
        for (int x = 0; x < NR; ++x) {
            std::vector<std::pair<int, int>> v;
            for (int i = 0; i < SHBASE / 2; ++i) for (int j = 0; j < 8; ++j) {
                const int pm = 2 * i + (x & 1), pn = 4 * j + (x >> 1);
                const int region = (pm < 16 ? 0 : 2) + (pn < 16 ? 0 : 1);
                v.push_back({region * 4096 + 1024 + j * 64 + i, pm | pn << 5});
            }
            for (int i = x; i < NOWN; i += NR) { const int pn = (own[i] >> 5) & 31; v.push_back({(pn < 16 ? 0 : 1) * 4096 + i, own[i] | (i + 1) << 10}); }
            std::sort(v.begin(), v.end());
            if ((int)v.size() != QTILES) { fprintf(stderr, "list size mismatch\n"); return; }
            for (int k = 0; k < QTILES; ++k) {
                const bool split = (k >= A0 && k < A0 + S0) || k >= QTILES - NSPLIT;
                if (split && (v[k].second >> 10)) { fprintf(stderr, "shared tile in a split segment\n"); return; }
                P.order[x][k] = (unsigned short)v[k].second;
            }
        }
        init = true;
    }
    P.x = (const float*)d_in[0]; P.w = (const float*)d_in[1]; P.out = (float*)d_out; P.rank = rank;
    for (int r = 0; r < NR; ++r) { P.ws[r] = (char*)d_ws_all[r]; P.outs[r] = (float*)d_out_all[r]; }
    hipLaunchKernelGGL(fused_kernel, dim3(GRID), dim3(512), LDS_BYTES, stream, P);
    HIP_CHECK(hipGetLastError());
}
```
